```python
import numpy as np

kernel_name = "gpu_gemm_a2a_m4096_k4096_n16384_w8a8_dequant"


ABSMAX_RTOL = 0.001

COMPUTE_DTYPE = "int8"
EPILOGUE = "dequant"
IN_QUANT = "int8"
OUT_QUANT = None
IN_DTYPES = {"x": "int8", "w_mat": "int8",
             "scale_x": "float32", "scale_w": "float32"}
WORLD_SIZE = 8

def setup_inputs():

    x = np.random.randint(-127, 128, size=(4096, 4096)).astype(np.int8)
    w_mat = np.random.randint(-127, 128, size=(4096, 16384)).astype(np.int8)
    scale_x = (0.02 * (1.0 + np.random.rand(1))).astype(np.float32)
    scale_w = (0.02 * (1.0 + np.random.rand(1))).astype(np.float32)
    return {"x": x, "w_mat": w_mat, "scale_x": scale_x, "scale_w": scale_w}

def reference(x, w_mat, scale_x, scale_w):
    acc = x.astype(np.float64) @ w_mat.astype(np.float64)
    y = (acc.astype(np.float32) * (scale_x * scale_w)).astype(np.float32)
    return y
```

```cpp
#include <hip/hip_runtime.h>
#include <rccl/rccl.h>
#include <cstdio>
#include <cstdint>

#define HIP_CHECK(expr) do { hipError_t _e = (expr); if (_e != hipSuccess) { \
    fprintf(stderr, "HIP error %s at %s:%d\n", hipGetErrorString(_e), __FILE__, __LINE__); \
    return; } } while (0)

#define LAS __attribute__((address_space(3)))
typedef int      i32x4 __attribute__((ext_vector_type(4)));
typedef int      i32x2 __attribute__((ext_vector_type(2)));
typedef unsigned u32x4 __attribute__((ext_vector_type(4)));
typedef float    f32x4 __attribute__((ext_vector_type(4)));

namespace ag {
constexpr int NR = 8;
constexpr int MSH = 512;
constexpr int KD = 4096;
constexpr int NFULL = 16384;
constexpr int NSH = 2048;
constexpr int NWG = 256;
constexpr int ASTG = 16384;
constexpr int BSTG = 8192;
#define AG_DA 7
#define AG_DB 6
constexpr int DA = AG_DA, DB = AG_DB;
constexpr int LDS_BYTES = DA * ASTG + DB * BSTG;
constexpr int NSTEP = 64;
constexpr int NTHREADS = 768;
constexpr size_t XBYTES = (size_t)MSH * KD;
constexpr size_t WS_CTR = 0;
constexpr size_t WS_TMO = 4096;
constexpr size_t WS_FLAGS = 8192;
constexpr size_t WS_X = 1u << 20;
constexpr size_t WS_NEED = WS_X + 2 * XBYTES;
constexpr unsigned long long DEADLINE = 50000000ull;

struct CA { static constexpr int value = DA, per = 8; };
struct CB { static constexpr int value = DB, per = 4; };

struct Params {
    const char* x; const char* w; const float* sx; const float* sw; float* out;
    char* ws[NR];
    int rank;
};

__device__ __forceinline__ void st16_sys(void* p, u32x4 v) {
    asm volatile("global_store_dwordx4 %0, %1, off sc0 sc1\n\ts_nop 1" :: "v"(p), "v"(v) : "memory");
}
__device__ __forceinline__ u32x4 ld16_sys_wait(const void* p) {
    u32x4 v;
    asm volatile("global_load_dwordx4 %0, %1, off sc0 sc1\n\ts_waitcnt vmcnt(0)" : "=&v"(v) : "v"(p) : "memory");
    return v;
}
#define AG_CST 1
__device__ __forceinline__ void st_out(float* p, f32x4 v) {
    asm volatile("global_store_dwordx4 %0, %1, off sc1\n\ts_nop 1" :: "v"(p), "v"(v) : "memory");
}
__device__ __forceinline__ const char* uniform_ptr(const char* p) {
    const unsigned lo = __builtin_amdgcn_readfirstlane((unsigned)(uintptr_t)p);
    const unsigned hi = __builtin_amdgcn_readfirstlane((unsigned)((uintptr_t)p >> 32));
    return (const char*)(((uintptr_t)hi << 32) | lo);
}
template <int N> __device__ __forceinline__ void waitvm_c() { asm volatile("s_waitcnt vmcnt(%0)" :: "n"(N) : "memory"); }
template <int PER, int K> __device__ __forceinline__ void waitvm_k(int k) {
    if constexpr (K <= 0) waitvm_c<0>();
    else { if (k >= K) waitvm_c<PER * K>(); else waitvm_k<PER, K - 1>(k); }
}
__device__ __forceinline__ int swzA(int row) { return (-(row >> 2)) & 3; }
__device__ __forceinline__ int swzB(int k) { return ((k >> 1) & 3) | (((k >> 4) & 1) << 2); }

__global__ __launch_bounds__(NTHREADS, 3) void fused_gather_gemm(Params P) {
    __shared__ __attribute__((aligned(1024))) char smem[LDS_BYTES];
    LAS char* L = (LAS char*)smem;
    const int tid = threadIdx.x;
    const int lane = tid & 63;
    const int wid = __builtin_amdgcn_readfirstlane(tid >> 6);
    const int b = blockIdx.x, g = b & 7, wi = b >> 3;
    const int rank = P.rank;
    const int src = (rank + g) & 7;
    char* wsl = P.ws[rank];
    uintptr_t wsp[NR];
#pragma unroll
    for (int r = 0; r < NR; ++r) { wsp[r] = (uintptr_t)P.ws[r]; asm volatile("" : "+s"(wsp[r])); }

    unsigned* ctr = (unsigned*)(wsl + WS_CTR) + b;
    const unsigned call = __hip_atomic_load(ctr, __ATOMIC_RELAXED, __HIP_MEMORY_SCOPE_AGENT) + 1u;
    if (tid < 512) {
        const int crow = 2 * b + (tid >> 8), cq = tid & 255;
        const u32x4 xv = *(const u32x4*)(P.x + (size_t)crow * KD + cq * 16);
        const int j = cq >> 2, c = cq & 3, r = crow & 255, rbk = crow >> 8;
        char* dst = wsl + WS_X + (size_t)(call & 1u) * XBYTES + (size_t)(rbk * NSTEP + j) * ASTG + r * 64 + ((c ^ swzA(r)) << 4);
        st16_sys(dst, xv);
    }
    asm volatile("s_waitcnt vmcnt(0)" ::: "memory");
    __builtin_amdgcn_s_barrier();
    asm volatile("" ::: "memory");
    if (tid < NR && tid != rank) {
        uintptr_t pw = wsp[0];
#pragma unroll
        for (int r = 1; r < NR; ++r) pw = (tid == r) ? wsp[r] : pw;
        __hip_atomic_store((unsigned*)((char*)pw + WS_FLAGS) + rank * NWG + b, call, __ATOMIC_RELAXED, __HIP_MEMORY_SCOPE_SYSTEM);
    }
    if (tid == 64) __hip_atomic_store(ctr, call, __ATOMIC_RELAXED, __HIP_MEMORY_SCOPE_AGENT);

    const int rb = wi >> 4, cb = wi & 15;
    const unsigned lds0 = (unsigned)(uintptr_t)L;
    unsigned* tmo = (unsigned*)(wsl + WS_TMO);
#define BAR() do { __builtin_amdgcn_sched_barrier(0); __builtin_amdgcn_s_barrier(); __builtin_amdgcn_sched_barrier(0); } while (0)
#define WAITL0() asm volatile("s_waitcnt lgkmcnt(0)" ::: "memory")

    if (wid >= 8) {
        const bool isA = wid < 10;
        const int lw = wid & 1;
        const char* gbase;
        unsigned voff[8];
        unsigned kstride;
        bool live = true;
        if (isA) {
            if (g == 0) {
                gbase = P.x; kstride = 64;
#pragma unroll
                for (int i = 0; i < 8; ++i) { const int row = lw * 128 + i * 16 + (lane >> 2); const int ch = (lane & 3) ^ swzA(row);
                    voff[i] = (unsigned)(row * KD + ch * 16 - (i & 3) * 1024 + 4096); }
            } else {
                gbase = P.ws[src] + WS_X + (size_t)(call & 1u) * XBYTES; kstride = ASTG;
#pragma unroll
                for (int i = 0; i < 8; ++i) voff[i] = (unsigned)(lw * 8192 + (i >> 2) * 4096 + lane * 16 + 4096);
            }
            gbase += (size_t)rb * (256 * KD) - 4096;
        } else {
            gbase = P.w + (size_t)rank * NSH + cb * 128 - 4096; kstride = 64u * NFULL;
#pragma unroll
            for (int i = 0; i < 8; ++i) { const int k = lw * 32 + (i & 3) * 8 + (lane >> 3); const int ch = (lane & 7) ^ swzB(k);
                voff[i] = (unsigned)(k * NFULL + ch * 16 - (i & 3) * 1024 + 4096); }
        }
        gbase = uniform_ptr(gbase);
        kstride = __builtin_amdgcn_readfirstlane(kstride);
        auto issueA = [&](int t, int slot) {
            const char* gp = uniform_ptr(gbase + (size_t)(unsigned)t * kstride);
            const unsigned lp = __builtin_amdgcn_readfirstlane(lds0 + (unsigned)(slot * ASTG + lw * 8192));
            unsigned keep;
            asm volatile("s_mov_b32 %0, m0\n\t"
                         "s_mov_b32 m0, %1\n\t"
                         "s_nop 4\n\t"
                         "global_load_lds_dwordx4 %2, %10\n\t"
                         "global_load_lds_dwordx4 %3, %10 offset:1024\n\t"
                         "global_load_lds_dwordx4 %4, %10 offset:2048\n\t"
                         "global_load_lds_dwordx4 %5, %10 offset:3072\n\t"
                         "s_add_u32 m0, m0, 0x1000\n\t"
                         "s_nop 0\n\t"
                         "global_load_lds_dwordx4 %6, %10\n\t"
                         "global_load_lds_dwordx4 %7, %10 offset:1024\n\t"
                         "global_load_lds_dwordx4 %8, %10 offset:2048\n\t"
                         "global_load_lds_dwordx4 %9, %10 offset:3072\n\t"
                         "s_mov_b32 m0, %0"
                         : "=&s"(keep)
                         : "s"(lp), "v"(voff[0]), "v"(voff[1]), "v"(voff[2]), "v"(voff[3]),
                           "v"(voff[4]), "v"(voff[5]), "v"(voff[6]), "v"(voff[7]), "s"(gp)
                         : "memory", "scc");
        };
        auto issueB = [&](int t, int slot) {
            const char* gp = uniform_ptr(gbase + (size_t)(unsigned)t * kstride);
            const unsigned lp = __builtin_amdgcn_readfirstlane(lds0 + (unsigned)(DA * ASTG + slot * BSTG + lw * 4096));
            unsigned keep;
            asm volatile("s_mov_b32 %0, m0\n\t"
                         "s_mov_b32 m0, %1\n\t"
                         "s_nop 4\n\t"
                         "global_load_lds_dwordx4 %2, %6\n\t"
                         "global_load_lds_dwordx4 %3, %6 offset:1024\n\t"
                         "global_load_lds_dwordx4 %4, %6 offset:2048\n\t"
                         "global_load_lds_dwordx4 %5, %6 offset:3072\n\t"
                         "s_mov_b32 m0, %0"
                         : "=&s"(keep)
                         : "s"(lp), "v"(voff[0]), "v"(voff[1]), "v"(voff[2]), "v"(voff[3]), "s"(gp)
                         : "memory");
        };
        if (isA) {
            if (g != 0) {
                const char* f = wsl + WS_FLAGS + (size_t)src * (NWG * 4) + lane * 16;
                const unsigned long long t0 = __builtin_amdgcn_s_memrealtime();
                for (unsigned it = 1;; ++it) {
                    const u32x4 v = ld16_sys_wait(f);
                    const bool ok = (int)(v.x - call) >= 0 && (int)(v.y - call) >= 0 && (int)(v.z - call) >= 0 && (int)(v.w - call) >= 0;
                    if (__all(ok)) break;
                    if ((it & 15u) == 0u) {
                        if (__hip_atomic_load(tmo, __ATOMIC_RELAXED, __HIP_MEMORY_SCOPE_AGENT) == call ||
                            __builtin_amdgcn_s_memrealtime() - t0 > DEADLINE) {
                            __hip_atomic_store(tmo, call, __ATOMIC_RELAXED, __HIP_MEMORY_SCOPE_AGENT);
                            asm volatile("s_waitcnt vmcnt(0)" ::: "memory");
                            live = false; break;
                        }
                    }
                    __builtin_amdgcn_s_sleep(1);
                }
            }
            if (live) {
#pragma unroll
                for (int t = 0; t < DA; ++t) issueA(t, t);
            }
            waitvm_c<8 * (DA - 1)>();
            BAR();
            int slot = 0;
            for (int t = 0; t < NSTEP; ++t) {
                waitvm_k<8, DA - 2>(NSTEP - 2 - t);
                BAR();
                if (live && t + DA < NSTEP) issueA(t + DA, slot);
                slot = (slot + 1 == DA) ? 0 : slot + 1;
            }
        } else {
#pragma unroll
            for (int t = 0; t < DB; ++t) issueB(t, t);
            waitvm_c<4 * (DB - 1)>();
            BAR();
            int slot = 0;
            for (int t = 0; t < NSTEP; ++t) {
                waitvm_k<4, DB - 2>(NSTEP - 2 - t);
                BAR();
                if (t + DB < NSTEP) issueB(t + DB, slot);
                slot = (slot + 1 == DB) ? 0 : slot + 1;
            }
        }
        asm volatile("s_waitcnt vmcnt(0)" ::: "memory");
        return;
    }

    const int wr = wid >> 1, wc = wid & 1, fr = lane & 15, fq = lane >> 4;
    float* orow = P.out + (size_t)(src * MSH + rb * 256 + wr * 64 + fr) * NSH + cb * 128 + wc * 64 + fq * 4;
    const unsigned aoff = (unsigned)((wr * 64 + fr) * 64 + ((fq ^ swzA(fr)) << 4));
    unsigned boff[4];
    {
        const int q = fr >> 1, p = fr & 1, sw = swzB(fq * 16 + q);
#pragma unroll
        for (int n = 0; n < 4; ++n) boff[n] = (unsigned)(DA * ASTG + (fq * 16 + q) * 128 + (((wc * 4 + n) ^ sw) << 4) + p * 8);
    }
    i32x4 acc[4][4];
#pragma unroll
    for (int m = 0; m < 4; ++m)
#pragma unroll
        for (int n = 0; n < 4; ++n) acc[m][n] = (i32x4){0, 0, 0, 0};
    i32x4 af[2][4], bf[2][4];
    auto rd = [&](int h, int sa, int sb) {
        const LAS char* pa = L + sa * ASTG;
        const LAS char* pb = L + sb * BSTG;
#pragma unroll
        for (int n = 0; n < 4; ++n) {
            const i32x2 lo = __builtin_amdgcn_ds_read_tr8_b64_v2i32((LAS i32x2*)(pb + boff[n]));
            const i32x2 hi = __builtin_amdgcn_ds_read_tr8_b64_v2i32((LAS i32x2*)(pb + boff[n] + 1024));
            bf[h][n] = (i32x4){lo.x, lo.y, hi.x, hi.y};
        }
#pragma unroll
        for (int m = 0; m < 4; ++m) af[h][m] = *(const LAS i32x4*)(pa + aoff + m * 1024);
    };
    auto mm = [&](int h) {
#pragma unroll
        for (int m = 0; m < 4; ++m)
#pragma unroll
            for (int n = 0; n < 4; ++n)
                acc[m][n] = __builtin_amdgcn_mfma_i32_16x16x64_i8(bf[h][n], af[h][m], acc[m][n], 0, 0, 0);
    };
#define INTERLEAVE() do { _Pragma("unroll") for (int _i = 0; _i < 12; ++_i) { \
        __builtin_amdgcn_sched_group_barrier(0x008, 1, 0); __builtin_amdgcn_sched_group_barrier(0x100, 1, 0); } \
        __builtin_amdgcn_sched_group_barrier(0x008, 4, 0); } while (0)
    BAR();
    const unsigned tmo_v = __hip_atomic_load(tmo, __ATOMIC_RELAXED, __HIP_MEMORY_SCOPE_AGENT);
    rd(0, 0, 0);
    int sa = 0, sb = 0;
    for (int t = 0; t < NSTEP; t += 2) {
#pragma unroll
        for (int h = 0; h < 2; ++h) {
            WAITL0();
            BAR();
            sa = (sa + 1 == DA) ? 0 : sa + 1;
            sb = (sb + 1 == DB) ? 0 : sb + 1;
            rd(h ^ 1, sa, sb);
            mm(h);
            INTERLEAVE();
            __builtin_amdgcn_sched_barrier(0);
        }
    }
    {
        const float sc = (tmo_v == call) ? __builtin_nanf("") : P.sx[0] * P.sw[0];
#pragma unroll
        for (int m = 0; m < 4; ++m)
#pragma unroll
            for (int n = 0; n < 4; ++n) {
                const i32x4 a = acc[m][n];
                const f32x4 v = {(float)a.x * sc, (float)a.y * sc, (float)a.z * sc, (float)a.w * sc};
                st_out(orow + (size_t)(m * 16) * NSH + n * 16, v);
            }
    }
}
}

extern "C" void kernel_launch(int rank, int nranks, ncclComm_t comm, hipStream_t stream,
                              void* const* d_in, const int* in_sizes, int n_in,
                              void* d_out, int out_size,
                              const int* in_dtypes, int out_dtype,
                              void* const* d_out_all, void* const* d_ws_all, size_t ws_bytes) {
    (void)comm; (void)in_dtypes; (void)out_dtype; (void)d_out_all;
    if (nranks != ag::NR || n_in != 4 || (unsigned)rank >= (unsigned)nranks || ws_bytes < ag::WS_NEED ||
        in_sizes[0] != ag::MSH * ag::KD || in_sizes[1] != ag::KD * ag::NFULL || out_size != ag::NR * ag::MSH * ag::NSH) {
        fprintf(stderr, "kernel_launch: unexpected problem shape\n");
        return;
    }
    ag::Params p;
    p.x = (const char*)d_in[0]; p.w = (const char*)d_in[1];
    p.sx = (const float*)d_in[2]; p.sw = (const float*)d_in[3];
    p.out = (float*)d_out;
    for (int r = 0; r < ag::NR; ++r) p.ws[r] = (char*)d_ws_all[r];
    p.rank = rank;
    hipLaunchKernelGGL(ag::fused_gather_gemm, dim3(ag::NWG), dim3(ag::NTHREADS), 0, stream, p);
    HIP_CHECK(hipGetLastError());
}
```

```cpp
#include <hip/hip_runtime.h>
#include <rccl/rccl.h>
#include <cstdio>
#include <cstdint>

#define HIP_CHECK(expr) do { hipError_t _e = (expr); if (_e != hipSuccess) { \
    fprintf(stderr, "HIP error %s at %s:%d\n", hipGetErrorString(_e), __FILE__, __LINE__); \
    return; } } while (0)

#define LAS __attribute__((address_space(3)))
#define GAS __attribute__((address_space(1)))
typedef int      i32x4 __attribute__((ext_vector_type(4)));
typedef int      i32x2 __attribute__((ext_vector_type(2)));
typedef unsigned u32x4 __attribute__((ext_vector_type(4)));
typedef float    f32x4 __attribute__((ext_vector_type(4)));

namespace ag {
constexpr int NR = 8;
constexpr int MSH = 512;
constexpr int KD = 4096;
constexpr int NFULL = 16384;
constexpr int NSH = 2048;
constexpr int NWG = 256;
constexpr int AUNIT = 8192;
constexpr int BSTG = 8192;
#define AG_DA 5
#define AG_NB 10
#define AG_S 5
constexpr int DA = AG_DA;
constexpr int NB = AG_NB;
constexpr int SKEW = AG_S;
constexpr int NK = 64;
constexpr int BOFF = 2 * DA * AUNIT;
constexpr int LDS_BYTES = BOFF + NB * BSTG;
constexpr int NTHREADS = 768;
#define AG_PAD 0
constexpr size_t XREG = (size_t)NK * AUNIT + AG_PAD;
constexpr size_t XBYTES = 4 * XREG;
constexpr size_t WS_CTR = 0;
constexpr size_t WS_TMO = 4096;
constexpr size_t WS_FLAGS = 8192;
constexpr size_t WS_X = 1u << 20;
constexpr size_t WS_NEED = WS_X + 2 * XBYTES;
constexpr unsigned long long DEADLINE = 50000000ull;
static_assert(DA >= 3 && DA <= 8 && SKEW >= 0 && NB - SKEW >= 3 && NB - 1 <= 15 && LDS_BYTES <= 163840, "layout");

struct Params {
    const char* x; const char* w; const float* sx; const float* sw; float* out;
    char* ws[NR];
    int rank;
};

__device__ __forceinline__ void st16_sys(void* p, u32x4 v) {
    asm volatile("global_store_dwordx4 %0, %1, off sc0 sc1\n\ts_nop 1" :: "v"(p), "v"(v) : "memory");
}
__device__ __forceinline__ unsigned ld4_sys_wait(const void* p) {
    unsigned v;
    asm volatile("global_load_dword %0, %1, off sc0 sc1\n\ts_waitcnt vmcnt(0)" : "=&v"(v) : "v"(p) : "memory");
    return v;
}
#define AG_CST 1
__device__ __forceinline__ void st_out(float* p, f32x4 v) {
    asm volatile("global_store_dwordx4 %0, %1, off sc1\n\ts_nop 1" :: "v"(p), "v"(v) : "memory");
}
__device__ __forceinline__ const char* uniform_ptr(const char* p) {
    const unsigned lo = __builtin_amdgcn_readfirstlane((unsigned)(uintptr_t)p);
    const unsigned hi = __builtin_amdgcn_readfirstlane((unsigned)((uintptr_t)p >> 32));
    return (const char*)(((uintptr_t)hi << 32) | lo);
}
template <int N> __device__ __forceinline__ void waitvm_c() { asm volatile("s_waitcnt vmcnt(%0)" :: "n"(N) : "memory"); }
template <int PER, int K> __device__ __forceinline__ void waitvm_k(int k) {
    if constexpr (K <= 0) waitvm_c<0>();
    else { if (k >= K) waitvm_c<PER * K>(); else waitvm_k<PER, K - 1>(k); }
}
#define AG_IL 1
__device__ __forceinline__ unsigned stage_off(int j, int o) {
    return (unsigned)((j >> 2) * 32768 + (o >> 10) * 4096 + (j & 3) * 1024 + (o & 1023));
}
__device__ __forceinline__ int swzA(int row) { return (-(row >> 2)) & 3; }
__device__ __forceinline__ int swzB(int k) { return ((k >> 1) & 3) | (((k >> 4) & 1) << 2); }

__device__ __forceinline__ bool poll_flags(const char* f, int lane, int n, unsigned call, unsigned* tmo) {
    const unsigned long long t0 = __builtin_amdgcn_s_memrealtime();
    for (unsigned it = 1;; ++it) {
        const unsigned v = ld4_sys_wait(f + lane * 4);
        const bool ok = lane >= n || (int)(v - call) >= 0;
        if (__all(ok)) break;
        if ((it & 15u) == 0u) {
            if (__hip_atomic_load(tmo, __ATOMIC_RELAXED, __HIP_MEMORY_SCOPE_AGENT) == call ||
                __builtin_amdgcn_s_memrealtime() - t0 > DEADLINE) {
                __hip_atomic_store(tmo, call, __ATOMIC_RELAXED, __HIP_MEMORY_SCOPE_AGENT);
                asm volatile("s_waitcnt vmcnt(0)" ::: "memory");
                return false;
            }
        }
        __builtin_amdgcn_s_sleep(1);
    }
    return true;
}

__global__ __launch_bounds__(NTHREADS, 3) void fused_gather_gemm(Params P) {
    __shared__ __attribute__((aligned(1024))) char smem[LDS_BYTES];
    LAS char* L = (LAS char*)smem;
    const int tid = threadIdx.x;
    const int lane = tid & 63;
    const int wid = __builtin_amdgcn_readfirstlane(tid >> 6);
    const int b = blockIdx.x, g = b & 7, wi = b >> 3;
    const int rank = P.rank;
    const int src = (rank + g) & 7;
    char* wsl = P.ws[rank];
    const int rb = wi >> 4, cb = wi & 15;
    const unsigned lds0 = (unsigned)(uintptr_t)L;
    unsigned* tmo = (unsigned*)(wsl + WS_TMO);
#define BAR() do { __builtin_amdgcn_sched_barrier(0); __builtin_amdgcn_s_barrier(); __builtin_amdgcn_sched_barrier(0); } while (0)
#define WAITL0() asm volatile("s_waitcnt lgkmcnt(0)" ::: "memory")

    if (wid >= 10) {
        const int lw = wid & 1;
        unsigned voff[4];
#pragma unroll
        for (int i = 0; i < 4; ++i) { const int k = lw * 32 + i * 8 + (lane >> 3); const int ch = (lane & 7) ^ swzB(k);
            voff[i] = (unsigned)(k * NFULL + ch * 16 - i * 1024 + 4096); }
        const char* gp = uniform_ptr(P.w + (size_t)rank * NSH + cb * 128 - 4096);
        unsigned lp = __builtin_amdgcn_readfirstlane(lds0 + (unsigned)(BOFF + lw * 4096));
        const unsigned lpend = lp + NB * BSTG;
        auto issueB = [&]() {
            unsigned keep;
            asm volatile("s_mov_b32 %0, m0\n\t"
                         "s_mov_b32 m0, %1\n\t"
                         "s_nop 4\n\t"
                         "global_load_lds_dwordx4 %2, %6\n\t"
                         "global_load_lds_dwordx4 %3, %6 offset:1024\n\t"
                         "global_load_lds_dwordx4 %4, %6 offset:2048\n\t"
                         "global_load_lds_dwordx4 %5, %6 offset:3072\n\t"
                         "s_mov_b32 m0, %0"
                         : "=&s"(keep)
                         : "s"(lp), "v"(voff[0]), "v"(voff[1]), "v"(voff[2]), "v"(voff[3]), "s"(gp)
                         : "memory");
            gp += 64u * NFULL;
            lp += BSTG; if (lp == lpend) lp -= NB * BSTG;
        };
        BAR();
#pragma unroll
        for (int t = 0; t < NB; ++t) issueB();
        waitvm_c<4 * (NB - 1)>();
        BAR();
        int issued = NB;
        for (int t = 0; t < NK - 1; ++t) {
            waitvm_k<4, NB - 1>(issued - 2 - t);
            BAR();
            if (t >= SKEW && issued < NK) { issueB(); ++issued; }
        }
        return;
    }
    if (wid >= 8) {
        const int lw = wid & 1;
        const int lag = lw ? SKEW : 0;
        const int npro = DA - lag > 0 ? DA - lag : 0;
        BAR();
        const unsigned call = __builtin_amdgcn_readfirstlane(*(volatile LAS unsigned*)(L + lw * (DA * AUNIT)));
        WAITL0();
        bool live = true;
        if (g != 0 && npro > 0) live = poll_flags(wsl + WS_FLAGS + (size_t)src * (NWG * 4) + (rb * 2 + lw) * 256, lane, 8 * ((npro + 7) / 8), call, tmo);
        unsigned voff[8];
        const char* gbase;
        if (g == 0) {
            gbase = P.x + (size_t)(rb * 256 + lw * 128) * KD;
#pragma unroll
            for (int i = 0; i < 8; ++i) { const int row = i * 16 + (lane >> 2); const int ch = (lane & 3) ^ swzA(row);
                voff[i] = (unsigned)(row * KD + ch * 16 - (i & 3) * 1024 + 4096); }
        } else {
            gbase = P.ws[src] + WS_X + (size_t)(call & 1u) * XBYTES + (size_t)(rb * 2 + lw) * XREG;
#pragma unroll
            for (int i = 0; i < 8; ++i) voff[i] = stage_off(0, i * 1024) + (unsigned)(lane * 16 - (i & 3) * 1024 + 4096);
        }
        gbase = uniform_ptr(gbase - 4096);
        int ju = 0;
        unsigned lp = __builtin_amdgcn_readfirstlane(lds0 + (unsigned)(lw * (DA * AUNIT)));
        const unsigned lpend = lp + DA * AUNIT;
        auto issueA = [&]() {
            const char* gp = uniform_ptr(gbase + (g == 0 ? (unsigned)(ju * 64) : stage_off(ju, 0)));
            unsigned keep;
            if (live)
            asm volatile("s_mov_b32 %0, m0\n\t"
                         "s_mov_b32 m0, %1\n\t"
                         "s_nop 4\n\t"
                         "global_load_lds_dwordx4 %2, %10\n\t"
                         "global_load_lds_dwordx4 %3, %10 offset:1024\n\t"
                         "global_load_lds_dwordx4 %4, %10 offset:2048\n\t"
                         "global_load_lds_dwordx4 %5, %10 offset:3072\n\t"
                         "s_add_u32 m0, m0, 0x1000\n\t"
                         "s_nop 0\n\t"
                         "global_load_lds_dwordx4 %6, %10\n\t"
                         "global_load_lds_dwordx4 %7, %10 offset:1024\n\t"
                         "global_load_lds_dwordx4 %8, %10 offset:2048\n\t"
                         "global_load_lds_dwordx4 %9, %10 offset:3072\n\t"
                         "s_mov_b32 m0, %0"
                         : "=&s"(keep)
                         : "s"(lp), "v"(voff[0]), "v"(voff[1]), "v"(voff[2]), "v"(voff[3]),
                           "v"(voff[4]), "v"(voff[5]), "v"(voff[6]), "v"(voff[7]), "s"(gp)
                         : "memory", "scc");
            ++ju;
            lp += AUNIT; if (lp == lpend) lp -= DA * AUNIT;
        };
        for (int u = 0; u < npro; ++u) issueA();
        int issued = npro;
        if (lag == 0) waitvm_k<8, DA - 1>(issued - 1);
        BAR();
        for (int t = 0; t < lag + NK - 1; ++t) {
            const int need = t + 1 - lag;
            if (need >= 0) waitvm_k<8, DA - 1>(issued - 1 - need);
            BAR();
            if (issued < NK && issued <= t - lag + DA) { issueA(); ++issued; }
        }
        return;
    }

    unsigned* ctr = (unsigned*)(wsl + WS_CTR) + b;
    const unsigned call = __hip_atomic_load(ctr, __ATOMIC_RELAXED, __HIP_MEMORY_SCOPE_AGENT) + 1u;
    float scale;
    {
        const int r = (b & 7) * 16 + (tid >> 5), q = tid & 31, j = ((b >> 3) & 7) * 8 + (q >> 2), c = q & 3;
        const u32x4 xv = *(const u32x4*)(P.x + (size_t)((b >> 6) * 128 + r) * KD + j * 64 + c * 16);
        scale = P.sx[0] * P.sw[0];
        st16_sys(wsl + WS_X + (size_t)(call & 1u) * XBYTES + (size_t)(b >> 6) * XREG + stage_off(j, r * 64 + ((c ^ swzA(r)) << 4)), xv);
        if (tid == 0) { *(volatile LAS unsigned*)(L) = call; *(volatile LAS unsigned*)(L + DA * AUNIT) = call; }
        asm volatile("s_waitcnt vmcnt(0) lgkmcnt(0)" : "+v"(scale) :: "memory");
    }
    BAR();
    if (tid < NR && tid != rank) {
        uintptr_t w0 = (uintptr_t)P.ws[0], w1 = (uintptr_t)P.ws[1], w2 = (uintptr_t)P.ws[2], w3 = (uintptr_t)P.ws[3];
        uintptr_t w4 = (uintptr_t)P.ws[4], w5 = (uintptr_t)P.ws[5], w6 = (uintptr_t)P.ws[6], w7 = (uintptr_t)P.ws[7];
        asm volatile("" : "+s"(w0), "+s"(w1), "+s"(w2), "+s"(w3), "+s"(w4), "+s"(w5), "+s"(w6), "+s"(w7));
        const uintptr_t pw = tid == 0 ? w0 : tid == 1 ? w1 : tid == 2 ? w2 : tid == 3 ? w3 : tid == 4 ? w4 : tid == 5 ? w5 : tid == 6 ? w6 : w7;
        __hip_atomic_store((GAS unsigned*)((GAS char*)pw + WS_FLAGS) + rank * NWG + b, call, __ATOMIC_RELAXED, __HIP_MEMORY_SCOPE_SYSTEM);
    }
    if (tid == 64) __hip_atomic_store(ctr, call, __ATOMIC_RELAXED, __HIP_MEMORY_SCOPE_AGENT);
    const bool lower = wid >= 4;
    if ((wid & 3) == 1 && g != 0) (void)poll_flags(wsl + WS_FLAGS + (size_t)src * (NWG * 4) + (rb * 2 + (lower ? 1 : 0)) * 256, lane, 64, call, tmo);

    const int wr = wid >> 1, wc = wid & 1, fr = lane & 15, fq = lane >> 4;
    const unsigned aoff = (unsigned)((lower ? DA * AUNIT : 0) + ((wr & 1) * 64 + fr) * 64 + ((fq ^ swzA(fr)) << 4));
    unsigned boff[4];
    {
        const int q = fr >> 1, p = fr & 1, sw = swzB(fq * 16 + q);
#pragma unroll
        for (int n = 0; n < 4; ++n) boff[n] = (unsigned)(BOFF + (fq * 16 + q) * 128 + (((wc * 4 + n) ^ sw) << 4) + p * 8);
    }
    i32x4 acc[4][4];
#pragma unroll
    for (int m = 0; m < 4; ++m)
#pragma unroll
        for (int n = 0; n < 4; ++n) acc[m][n] = (i32x4){0, 0, 0, 0};
    i32x4 af[2][4], bf[2][4];
    auto rd = [&](int h, int sa, int sb) {
        const LAS char* pa = L + sa * AUNIT;
        const LAS char* pb = L + sb * BSTG;
#pragma unroll
        for (int n = 0; n < 4; ++n) {
            const i32x2 lo = __builtin_amdgcn_ds_read_tr8_b64_v2i32((LAS i32x2*)(pb + boff[n]));
            const i32x2 hi = __builtin_amdgcn_ds_read_tr8_b64_v2i32((LAS i32x2*)(pb + boff[n] + 1024));
            bf[h][n] = (i32x4){lo.x, lo.y, hi.x, hi.y};
        }
#pragma unroll
        for (int m = 0; m < 4; ++m) af[h][m] = *(const LAS i32x4*)(pa + aoff + m * 1024);
    };
    auto mm = [&](int h) {
#pragma unroll
        for (int m = 0; m < 4; ++m)
#pragma unroll
            for (int n = 0; n < 4; ++n)
                acc[m][n] = __builtin_amdgcn_mfma_i32_16x16x64_i8(bf[h][n], af[h][m], acc[m][n], 0, 0, 0);
    };
#define INTERLEAVE() do { _Pragma("unroll") for (int _i = 0; _i < 12; ++_i) { \
        __builtin_amdgcn_sched_group_barrier(0x008, 1, 0); __builtin_amdgcn_sched_group_barrier(0x100, 1, 0); } \
        __builtin_amdgcn_sched_group_barrier(0x008, 4, 0); } while (0)
    BAR();
    const unsigned tmo_v = __hip_atomic_load(tmo, __ATOMIC_RELAXED, __HIP_MEMORY_SCOPE_AGENT);
    if (lower) for (int t = 0; t < SKEW; ++t) BAR();
    int sa = 0, sb = 0;
    rd(0, 0, 0);
    for (int u = 0; u < NK; u += 2) {
#pragma unroll
        for (int h = 0; h < 2; ++h) {
            WAITL0();
            BAR();
            sa = (sa + 1 == DA) ? 0 : sa + 1;
            sb = (sb + 1 == NB) ? 0 : sb + 1;
            rd(h ^ 1, sa, sb);
            mm(h);
            INTERLEAVE();
            __builtin_amdgcn_sched_barrier(0);
        }
    }
    {
        int tz = tid; asm volatile("" : "+v"(tz));
        const int ln = tz & 63;
        float* orow = P.out + (size_t)(src * MSH + rb * 256 + wr * 64 + (ln & 15)) * NSH + cb * 128 + wc * 64 + (ln >> 4) * 4;
        unsigned tv = tmo_v; asm volatile("" : "+v"(tv));
        const float sc = (tv == call) ? __builtin_nanf("") : scale;
#pragma unroll
        for (int m = 0; m < 4; ++m)
#pragma unroll
            for (int n = 0; n < 4; ++n) {
                const i32x4 a = acc[m][n];
                const f32x4 v = {(float)a.x * sc, (float)a.y * sc, (float)a.z * sc, (float)a.w * sc};
                st_out(orow + (size_t)(m * 16) * NSH + n * 16, v);
            }
    }
}
}

extern "C" void kernel_launch(int rank, int nranks, ncclComm_t comm, hipStream_t stream,
                              void* const* d_in, const int* in_sizes, int n_in,
                              void* d_out, int out_size,
                              const int* in_dtypes, int out_dtype,
                              void* const* d_out_all, void* const* d_ws_all, size_t ws_bytes) {
    (void)comm; (void)in_dtypes; (void)out_dtype; (void)d_out_all;
    if (nranks != ag::NR || n_in != 4 || (unsigned)rank >= (unsigned)nranks || ws_bytes < ag::WS_NEED ||
        in_sizes[0] != ag::MSH * ag::KD || in_sizes[1] != ag::KD * ag::NFULL || out_size != ag::NR * ag::MSH * ag::NSH) {
        fprintf(stderr, "kernel_launch: unexpected problem shape\n");
        return;
    }
    ag::Params p;
    p.x = (const char*)d_in[0]; p.w = (const char*)d_in[1];
    p.sx = (const float*)d_in[2]; p.sw = (const float*)d_in[3];
    p.out = (float*)d_out;
    for (int r = 0; r < ag::NR; ++r) p.ws[r] = (char*)d_ws_all[r];
    p.rank = rank;
    hipLaunchKernelGGL(ag::fused_gather_gemm, dim3(ag::NWG), dim3(ag::NTHREADS), 0, stream, p);
    HIP_CHECK(hipGetLastError());
}
```

```cpp
#include <hip/hip_runtime.h>
#include <rccl/rccl.h>
#include <cstdio>
#include <cstdint>

#define HIP_CHECK(expr) do { hipError_t _e = (expr); if (_e != hipSuccess) { \
    fprintf(stderr, "HIP error %s at %s:%d\n", hipGetErrorString(_e), __FILE__, __LINE__); \
    return; } } while (0)

#define LAS __attribute__((address_space(3)))
#define GAS __attribute__((address_space(1)))
typedef int      i32x4 __attribute__((ext_vector_type(4)));
typedef int      i32x2 __attribute__((ext_vector_type(2)));
typedef unsigned u32x4 __attribute__((ext_vector_type(4)));
typedef float    f32x4 __attribute__((ext_vector_type(4)));

namespace ag {
constexpr int NR = 8;
constexpr int MSH = 512;
constexpr int KD = 4096;
constexpr int NFULL = 16384;
constexpr int NSH = 2048;
constexpr int NWG = 256;
constexpr int AUNIT = 8192;
constexpr int BSTG = 8192;
#define AG_DA 5
#define AG_NB 10
#define AG_S 5
constexpr int DA = AG_DA;
constexpr int NB = AG_NB;
constexpr int SKEW = AG_S;
constexpr int NK = 64;
constexpr int BOFF = 2 * DA * AUNIT;
constexpr int LDS_BYTES = BOFF + NB * BSTG;
constexpr int NTHREADS = 768;
#define AG_PAD 0
constexpr size_t XREG = (size_t)NK * AUNIT + AG_PAD;
constexpr size_t XBYTES = 4 * XREG;
constexpr size_t WS_CTR = 0;
constexpr size_t WS_TMO = 4096;
constexpr size_t WS_TICK = 4096 + 1024;
constexpr size_t WS_FLAGS = 8192;
constexpr size_t WS_X = 1u << 20;
constexpr size_t WS_NEED = WS_X + 2 * XBYTES;
constexpr unsigned long long DEADLINE = 50000000ull;
static_assert(DA >= 3 && DA <= 8 && SKEW >= 0 && NB - SKEW >= 3 && NB - 1 <= 15 && LDS_BYTES <= 163840, "layout");

struct Params {
    const char* x; const char* w; const float* sx; const float* sw; float* out;
    char* ws[NR];
    int rank;
};

__device__ __forceinline__ void st16_sys(void* p, u32x4 v) {
    asm volatile("global_store_dwordx4 %0, %1, off sc0 sc1\n\ts_nop 1" :: "v"(p), "v"(v) : "memory");
}
__device__ __forceinline__ unsigned ld4_sys_wait(const void* p) {
    unsigned v;
    asm volatile("global_load_dword %0, %1, off sc0 sc1\n\ts_waitcnt vmcnt(0)" : "=&v"(v) : "v"(p) : "memory");
    return v;
}
__device__ __forceinline__ unsigned ld4_agent_wait(const void* p) {
    unsigned v;
    asm volatile("global_load_dword %0, %1, off sc1\n\ts_waitcnt vmcnt(0)" : "=&v"(v) : "v"(p) : "memory");
    return v;
}
#define AG_CST 1
__device__ __forceinline__ void st_out(float* p, f32x4 v) {
    asm volatile("global_store_dwordx4 %0, %1, off sc1\n\ts_nop 1" :: "v"(p), "v"(v) : "memory");
}
__device__ __forceinline__ const char* uniform_ptr(const char* p) {
    const unsigned lo = __builtin_amdgcn_readfirstlane((unsigned)(uintptr_t)p);
    const unsigned hi = __builtin_amdgcn_readfirstlane((unsigned)((uintptr_t)p >> 32));
    return (const char*)(((uintptr_t)hi << 32) | lo);
}
template <int N> __device__ __forceinline__ void waitvm_c() { asm volatile("s_waitcnt vmcnt(%0)" :: "n"(N) : "memory"); }
template <int PER, int K> __device__ __forceinline__ void waitvm_k(int k) {
    if constexpr (K <= 0) waitvm_c<0>();
    else { if (k >= K) waitvm_c<PER * K>(); else waitvm_k<PER, K - 1>(k); }
}
#define AG_IL 1
__device__ __forceinline__ unsigned stage_off(int j, int o) {
    return (unsigned)((j >> 2) * 32768 + (o >> 10) * 4096 + (j & 3) * 1024 + (o & 1023));
}
__device__ __forceinline__ int swzA(int row) { return (-(row >> 2)) & 3; }
__device__ __forceinline__ int swzB(int k) { return ((k >> 1) & 3) | (((k >> 4) & 1) << 2); }

__device__ __forceinline__ bool poll_flags(const char* f, int lane, int n, unsigned call, unsigned* tmo) {
    const unsigned long long t0 = __builtin_amdgcn_s_memrealtime();
    for (unsigned it = 1;; ++it) {
        const unsigned v = ld4_sys_wait(f + lane * 4);
        const bool ok = lane >= n || (int)(v - call) >= 0;
        if (__all(ok)) break;
        if ((it & 15u) == 0u) {
            if (__hip_atomic_load(tmo, __ATOMIC_RELAXED, __HIP_MEMORY_SCOPE_AGENT) == call ||
                __builtin_amdgcn_s_memrealtime() - t0 > DEADLINE) {
                __hip_atomic_store(tmo, call, __ATOMIC_RELAXED, __HIP_MEMORY_SCOPE_AGENT);
                asm volatile("s_waitcnt vmcnt(0)" ::: "memory");
                return false;
            }
        }
#define AG_SLEEP 1
        __builtin_amdgcn_s_sleep(AG_SLEEP);
    }
    return true;
}

__global__ __launch_bounds__(NTHREADS, 3) void fused_gather_gemm(Params P) {
    __shared__ __attribute__((aligned(1024))) char smem[LDS_BYTES];
    LAS char* L = (LAS char*)smem;
    const int tid = threadIdx.x;
    const int lane = tid & 63;
    const int wid = __builtin_amdgcn_readfirstlane(tid >> 6);
    const int b = blockIdx.x;
    const int rank = P.rank;
    char* wsl = P.ws[rank];
    const unsigned lds0 = (unsigned)(uintptr_t)L;
    unsigned* tmo = (unsigned*)(wsl + WS_TMO);
#define BAR() do { __builtin_amdgcn_sched_barrier(0); __builtin_amdgcn_s_barrier(); __builtin_amdgcn_sched_barrier(0); } while (0)
#define WAITL0() asm volatile("s_waitcnt lgkmcnt(0)" ::: "memory")

    volatile LAS unsigned* mail = (volatile LAS unsigned*)(L + BOFF + (NB - 1) * BSTG + BSTG - 16);
    if (wid >= 10) {
        const int lw = wid & 1;
        unsigned voff[4];
#pragma unroll
        for (int i = 0; i < 4; ++i) { const int k = lw * 32 + i * 8 + (lane >> 3); const int ch = (lane & 7) ^ swzB(k);
            voff[i] = (unsigned)(k * NFULL + ch * 16 - i * 1024 + 4096); }
        BAR();
        BAR();
        const int cb = (int)__builtin_amdgcn_readfirstlane(mail[2]) & 15;
        WAITL0();
        BAR();
        const char* gp = uniform_ptr(P.w + (size_t)rank * NSH + cb * 128 - 4096);
        unsigned lp = __builtin_amdgcn_readfirstlane(lds0 + (unsigned)(BOFF + lw * 4096));
        const unsigned lpend = lp + NB * BSTG;
        auto issueB = [&]() {
            unsigned keep;
            asm volatile("s_mov_b32 %0, m0\n\t"
                         "s_mov_b32 m0, %1\n\t"
                         "s_nop 4\n\t"
                         "global_load_lds_dwordx4 %2, %6\n\t"
                         "global_load_lds_dwordx4 %3, %6 offset:1024\n\t"
                         "global_load_lds_dwordx4 %4, %6 offset:2048\n\t"
                         "global_load_lds_dwordx4 %5, %6 offset:3072\n\t"
                         "s_mov_b32 m0, %0"
                         : "=&s"(keep)
                         : "s"(lp), "v"(voff[0]), "v"(voff[1]), "v"(voff[2]), "v"(voff[3]), "s"(gp)
                         : "memory");
            gp += 64u * NFULL;
            lp += BSTG; if (lp == lpend) lp -= NB * BSTG;
        };
#pragma unroll
        for (int t = 0; t < NB; ++t) issueB();
        waitvm_c<4 * (NB - 1)>();
        BAR();
        int issued = NB;
        for (int t = 0; t < NK - 1; ++t) {
            waitvm_k<4, NB - 1>(issued - 2 - t);
            BAR();
            if (t >= SKEW && issued < NK) { issueB(); ++issued; }
        }
        return;
    }
    if (wid >= 8) {
        const int lw = wid & 1;
        const int lag = lw ? SKEW : 0;
        const int npro = DA - lag > 0 ? DA - lag : 0;
        unsigned ticket = 1; int pref = 0;
        GAS unsigned* tick = (GAS unsigned*)(wsl + WS_TICK);
        if (lw == 0) {
            unsigned xcc; asm volatile("s_getreg_b32 %0, hwreg(HW_REG_XCC_ID)" : "=s"(xcc));
            const unsigned cx = 0x6320u, cy = 0x7541u;
            const bool inx = (0x4Du >> rank) & 1u;
            const unsigned own = inx ? cx : cy, oth = inx ? cy : cx;
            pref = (int)((((xcc & 4u) ? oth : own) >> ((xcc & 3u) * 4u)) & 7u);
            if (lane == 0) asm volatile("global_atomic_add %0, %1, %0, off sc0" : "+v"(ticket) : "v"(tick + pref * 32) : "memory");
        }
#define AG_WARM 0
        unsigned warm = 0;
        if (AG_WARM && lw == 1 && lane < 10) {
            const char* pw = lane == 0 ? P.ws[0] : lane == 1 ? P.ws[1] : lane == 2 ? P.ws[2] : lane == 3 ? P.ws[3] : lane == 4 ? P.ws[4] : lane == 5 ? P.ws[5] : lane == 6 ? P.ws[6] : P.ws[7];
            const char* a = lane < NR ? pw + 65536 + (rank * NWG + b) * 128
                                      : wsl + WS_X + (size_t)(lane - NR) * XBYTES + (size_t)(b >> 6) * XREG + (b & 63) * 4096;
            if ((lane < NR && (AG_WARM & 2)) || (lane >= NR && (AG_WARM & 1)))
                asm volatile("global_load_dword %0, %1, off sc0 sc1" : "+v"(warm) : "v"(a) : "memory");
        }
        BAR();
        const unsigned call = __builtin_amdgcn_readfirstlane(mail[0]);
        WAITL0();
        if (lw == 0) {
            asm volatile("s_waitcnt vmcnt(0)" : "+v"(ticket) :: "memory");
            const unsigned base = (call - 1u) * 32u;
            unsigned t = __builtin_amdgcn_readfirstlane(ticket) - base;
            int s = pref;
            for (int k = 1; t >= 32u && k < NR; ++k) {
                if (lane == 0) __hip_atomic_fetch_sub(tick + s * 32, 1u, __ATOMIC_RELAXED, __HIP_MEMORY_SCOPE_AGENT);
                s = (pref + k) & 7;
                unsigned v = 0;
                if (lane == 0) v = __hip_atomic_fetch_add(tick + s * 32, 1u, __ATOMIC_RELAXED, __HIP_MEMORY_SCOPE_AGENT);
                t = __builtin_amdgcn_readfirstlane(v) - base;
            }
            if (lane == 0) { mail[1] = (unsigned)s; mail[2] = t & 31u; }
            WAITL0();
        }
        BAR();
        const int src = (int)__builtin_amdgcn_readfirstlane(mail[1]) & 7, wi = (int)__builtin_amdgcn_readfirstlane(mail[2]) & 31;
        WAITL0();
        BAR();
        asm volatile("s_waitcnt vmcnt(0)" : "+v"(warm) :: "memory");
        const int rb = wi >> 4;
        const bool local = src == rank;
        bool live = true;
        if (!local && npro > 0) live = poll_flags(wsl + WS_FLAGS + (size_t)src * (NWG * 4) + (rb * 2 + lw) * 256, lane, 8 * ((npro + 7) / 8), call, tmo);
        unsigned voff[8];
        const char* gbase;
        if (local) {
            gbase = P.x + (size_t)(rb * 256 + lw * 128) * KD;
#pragma unroll
            for (int i = 0; i < 8; ++i) { const int row = i * 16 + (lane >> 2); const int ch = (lane & 3) ^ swzA(row);
                voff[i] = (unsigned)(row * KD + ch * 16 - (i & 3) * 1024 + 4096); }
        } else {
            gbase = P.ws[src] + WS_X + (size_t)(call & 1u) * XBYTES + (size_t)(rb * 2 + lw) * XREG;
#pragma unroll
            for (int i = 0; i < 8; ++i) voff[i] = stage_off(0, i * 1024) + (unsigned)(lane * 16 - (i & 3) * 1024 + 4096);
        }
        gbase = uniform_ptr(gbase - 4096);
        int ju = 0;
        unsigned lp = __builtin_amdgcn_readfirstlane(lds0 + (unsigned)(lw * (DA * AUNIT)));
        const unsigned lpend = lp + DA * AUNIT;
        auto issueA = [&]() {
            const char* gp = uniform_ptr(gbase + (local ? (unsigned)(ju * 64) : stage_off(ju, 0)));
            unsigned keep;
            if (live)
            asm volatile("s_mov_b32 %0, m0\n\t"
                         "s_mov_b32 m0, %1\n\t"
                         "s_nop 4\n\t"
                         "global_load_lds_dwordx4 %2, %10\n\t"
                         "global_load_lds_dwordx4 %3, %10 offset:1024\n\t"
                         "global_load_lds_dwordx4 %4, %10 offset:2048\n\t"
                         "global_load_lds_dwordx4 %5, %10 offset:3072\n\t"
                         "s_add_u32 m0, m0, 0x1000\n\t"
                         "s_nop 0\n\t"
                         "global_load_lds_dwordx4 %6, %10\n\t"
                         "global_load_lds_dwordx4 %7, %10 offset:1024\n\t"
                         "global_load_lds_dwordx4 %8, %10 offset:2048\n\t"
                         "global_load_lds_dwordx4 %9, %10 offset:3072\n\t"
                         "s_mov_b32 m0, %0"
                         : "=&s"(keep)
                         : "s"(lp), "v"(voff[0]), "v"(voff[1]), "v"(voff[2]), "v"(voff[3]),
                           "v"(voff[4]), "v"(voff[5]), "v"(voff[6]), "v"(voff[7]), "s"(gp)
                         : "memory", "scc");
            ++ju;
            lp += AUNIT; if (lp == lpend) lp -= DA * AUNIT;
        };
        for (int u = 0; u < npro; ++u) issueA();
        int issued = npro;
        if (lag == 0) waitvm_k<8, DA - 1>(issued - 1);
        BAR();
        for (int t = 0; t < lag + NK - 1; ++t) {
            const int need = t + 1 - lag;
            if (need >= 0) waitvm_k<8, DA - 1>(issued - 1 - need);
            BAR();
            if (issued < NK && issued <= t - lag + DA) { issueA(); ++issued; }
        }
        return;
    }

    unsigned* ctr = (unsigned*)(wsl + WS_CTR) + b;
    const int xr = (b & 7) * 16 + (tid >> 5), xq = tid & 31, xj = ((b >> 3) & 7) * 8 + (xq >> 2), xc = xq & 3;
    const u32x4 xv = *(const u32x4*)(P.x + (size_t)((b >> 6) * 128 + xr) * KD + xj * 64 + xc * 16);
    const unsigned call = __hip_atomic_load(ctr, __ATOMIC_RELAXED, __HIP_MEMORY_SCOPE_AGENT) + 1u;
    asm volatile("" ::: "memory");
    uintptr_t w0 = (uintptr_t)P.ws[0], w1 = (uintptr_t)P.ws[1], w2 = (uintptr_t)P.ws[2], w3 = (uintptr_t)P.ws[3];
    uintptr_t w4 = (uintptr_t)P.ws[4], w5 = (uintptr_t)P.ws[5], w6 = (uintptr_t)P.ws[6], w7 = (uintptr_t)P.ws[7];
    asm volatile("" : "+s"(w0), "+s"(w1), "+s"(w2), "+s"(w3), "+s"(w4), "+s"(w5), "+s"(w6), "+s"(w7));
    const uintptr_t pw = tid == 0 ? w0 : tid == 1 ? w1 : tid == 2 ? w2 : tid == 3 ? w3 : tid == 4 ? w4 : tid == 5 ? w5 : tid == 6 ? w6 : w7;
    st16_sys(wsl + WS_X + (size_t)(call & 1u) * XBYTES + (size_t)(b >> 6) * XREG + stage_off(xj, xr * 64 + ((xc ^ swzA(xr)) << 4)), xv);
    if (tid == 0) mail[0] = call;
    asm volatile("s_waitcnt vmcnt(0) lgkmcnt(0)" ::: "memory");
    BAR();
    if (tid < NR && tid != rank)
        __hip_atomic_store((GAS unsigned*)((GAS char*)pw + WS_FLAGS) + rank * NWG + b, call, __ATOMIC_RELAXED, __HIP_MEMORY_SCOPE_SYSTEM);
    if (tid == 64) __hip_atomic_store(ctr, call, __ATOMIC_RELAXED, __HIP_MEMORY_SCOPE_AGENT);
    float scale = P.sx[0] * P.sw[0];
    BAR();
    const int src = (int)__builtin_amdgcn_readfirstlane(mail[1]) & 7, wi = (int)__builtin_amdgcn_readfirstlane(mail[2]) & 31;
    WAITL0();
    BAR();
    const int rb = wi >> 4, cb = wi & 15;
    const bool lower = wid >= 4;
    if ((wid & 3) == 1 && src != rank) (void)poll_flags(wsl + WS_FLAGS + (size_t)src * (NWG * 4) + (rb * 2 + (lower ? 1 : 0)) * 256, lane, 64, call, tmo);

    const int wr = wid >> 1, wc = wid & 1, fr = lane & 15, fq = lane >> 4;
    const unsigned aoff = (unsigned)((lower ? DA * AUNIT : 0) + ((wr & 1) * 64 + fr) * 64 + ((fq ^ swzA(fr)) << 4));
    unsigned boff[4];
    {
        const int q = fr >> 1, p = fr & 1, sw = swzB(fq * 16 + q);
#pragma unroll
        for (int n = 0; n < 4; ++n) boff[n] = (unsigned)(BOFF + (fq * 16 + q) * 128 + (((wc * 4 + n) ^ sw) << 4) + p * 8);
    }
    i32x4 acc[4][4];
#pragma unroll
    for (int m = 0; m < 4; ++m)
#pragma unroll
        for (int n = 0; n < 4; ++n) acc[m][n] = (i32x4){0, 0, 0, 0};
    i32x4 af[2][4], bf[2][4];
    auto rd = [&](int h, int sa, int sb) {
        const LAS char* pa = L + sa * AUNIT;
        const LAS char* pb = L + sb * BSTG;
#pragma unroll
        for (int n = 0; n < 4; ++n) {
            const i32x2 lo = __builtin_amdgcn_ds_read_tr8_b64_v2i32((LAS i32x2*)(pb + boff[n]));
            const i32x2 hi = __builtin_amdgcn_ds_read_tr8_b64_v2i32((LAS i32x2*)(pb + boff[n] + 1024));
            bf[h][n] = (i32x4){lo.x, lo.y, hi.x, hi.y};
        }
#pragma unroll
        for (int m = 0; m < 4; ++m) af[h][m] = *(const LAS i32x4*)(pa + aoff + m * 1024);
    };
    auto mm = [&](int h) {
#pragma unroll
        for (int m = 0; m < 4; ++m)
#pragma unroll
            for (int n = 0; n < 4; ++n)
                acc[m][n] = __builtin_amdgcn_mfma_i32_16x16x64_i8(bf[h][n], af[h][m], acc[m][n], 0, 0, 0);
    };
#define INTERLEAVE() do { _Pragma("unroll") for (int _i = 0; _i < 12; ++_i) { \
        __builtin_amdgcn_sched_group_barrier(0x008, 1, 0); __builtin_amdgcn_sched_group_barrier(0x100, 1, 0); } \
        __builtin_amdgcn_sched_group_barrier(0x008, 4, 0); } while (0)
    asm volatile("" : "+v"(scale));
    BAR();
    const unsigned tmo_v = __hip_atomic_load(tmo, __ATOMIC_RELAXED, __HIP_MEMORY_SCOPE_AGENT);
    if (lower) for (int t = 0; t < SKEW; ++t) BAR();
    int sa = 0, sb = 0;
    rd(0, 0, 0);
    for (int u = 0; u < NK; u += 2) {
#pragma unroll
        for (int h = 0; h < 2; ++h) {
            WAITL0();
            BAR();
            sa = (sa + 1 == DA) ? 0 : sa + 1;
            sb = (sb + 1 == NB) ? 0 : sb + 1;
            rd(h ^ 1, sa, sb);
            mm(h);
            INTERLEAVE();
            __builtin_amdgcn_sched_barrier(0);
        }
    }
    {
        int tz = tid; asm volatile("" : "+v"(tz));
        const int ln = tz & 63;
        float* orow = P.out + (size_t)(src * MSH + rb * 256 + wr * 64 + (ln & 15)) * NSH + cb * 128 + wc * 64 + (ln >> 4) * 4;
        unsigned tv = tmo_v; asm volatile("" : "+v"(tv));
        const float sc = (tv == call) ? __builtin_nanf("") : scale;
#pragma unroll
        for (int m = 0; m < 4; ++m)
#pragma unroll
            for (int n = 0; n < 4; ++n) {
                const i32x4 a = acc[m][n];
                const f32x4 v = {(float)a.x * sc, (float)a.y * sc, (float)a.z * sc, (float)a.w * sc};
                st_out(orow + (size_t)(m * 16) * NSH + n * 16, v);
            }
    }
}
}

extern "C" void kernel_launch(int rank, int nranks, ncclComm_t comm, hipStream_t stream,
                              void* const* d_in, const int* in_sizes, int n_in,
                              void* d_out, int out_size,
                              const int* in_dtypes, int out_dtype,
                              void* const* d_out_all, void* const* d_ws_all, size_t ws_bytes) {
    (void)comm; (void)in_dtypes; (void)out_dtype; (void)d_out_all;
    if (nranks != ag::NR || n_in != 4 || (unsigned)rank >= (unsigned)nranks || ws_bytes < ag::WS_NEED ||
        in_sizes[0] != ag::MSH * ag::KD || in_sizes[1] != ag::KD * ag::NFULL || out_size != ag::NR * ag::MSH * ag::NSH) {
        fprintf(stderr, "kernel_launch: unexpected problem shape\n");
        return;
    }
    ag::Params p;
    p.x = (const char*)d_in[0]; p.w = (const char*)d_in[1];
    p.sx = (const float*)d_in[2]; p.sw = (const float*)d_in[3];
    p.out = (float*)d_out;
    for (int r = 0; r < ag::NR; ++r) p.ws[r] = (char*)d_ws_all[r];
    p.rank = rank;
    hipLaunchKernelGGL(ag::fused_gather_gemm, dim3(ag::NWG), dim3(ag::NTHREADS), 0, stream, p);
    HIP_CHECK(hipGetLastError());
}
```

```cpp
#include <hip/hip_runtime.h>
#include <rccl/rccl.h>
#include <cstdio>
#include <cstdint>

#define HIP_CHECK(expr) do { hipError_t _e = (expr); if (_e != hipSuccess) { \
    fprintf(stderr, "HIP error %s at %s:%d\n", hipGetErrorString(_e), __FILE__, __LINE__); \
    return; } } while (0)

#define LAS __attribute__((address_space(3)))
#define GAS __attribute__((address_space(1)))
typedef int      i32x4 __attribute__((ext_vector_type(4)));
typedef int      i32x2 __attribute__((ext_vector_type(2)));
typedef unsigned u32x4 __attribute__((ext_vector_type(4)));
typedef float    f32x4 __attribute__((ext_vector_type(4)));

namespace ag {
constexpr int NR = 8;
constexpr int MSH = 512;
constexpr int KD = 4096;
constexpr int NFULL = 16384;
constexpr int NSH = 2048;
constexpr int NWG = 256;
constexpr int AUNIT = 8192;
constexpr int BSTG = 8192;
#define AG_DA 5
#define AG_NB 10
#define AG_S 5
constexpr int DA = AG_DA;
constexpr int NB = AG_NB;
constexpr int SKEW = AG_S;
constexpr int NK = 64;
constexpr int BOFF = 2 * DA * AUNIT;
constexpr int LDS_BYTES = BOFF + NB * BSTG;
constexpr int NTHREADS = 768;
constexpr size_t XREG = (size_t)NK * AUNIT;
constexpr size_t XBYTES = 4 * XREG;
constexpr size_t WS_CTR = 0;
constexpr size_t WS_TMO = 4096;
constexpr size_t WS_TICK = 4096 + 1024;
constexpr size_t WS_FLAGS = 8192;
constexpr size_t WS_X = 1u << 20;
constexpr size_t WS_NEED = WS_X + 2 * XBYTES;
constexpr unsigned long long DEADLINE = 50000000ull;
static_assert(DA >= 3 && DA <= 8 && SKEW >= 0 && NB - SKEW >= 3 && NB - 1 <= 15 && LDS_BYTES <= 163840, "layout");

struct Params {
    const char* x; const char* w; const float* sx; const float* sw; float* out;
    char* ws[NR];
    int rank;
};

__device__ __forceinline__ void st16_sys(void* p, u32x4 v) {
    asm volatile("global_store_dwordx4 %0, %1, off sc0 sc1\n\ts_nop 1" :: "v"(p), "v"(v) : "memory");
}
__device__ __forceinline__ void st16_out(float* p, f32x4 v) {
    asm volatile("global_store_dwordx4 %0, %1, off sc1\n\ts_nop 1" :: "v"(p), "v"(v) : "memory");
}
__device__ __forceinline__ unsigned ld4_sys_wait(const void* p) {
    unsigned v;
    asm volatile("global_load_dword %0, %1, off sc0 sc1\n\ts_waitcnt vmcnt(0)" : "=&v"(v) : "v"(p) : "memory");
    return v;
}
__device__ __forceinline__ const char* uniform_ptr(const char* p) {
    const unsigned lo = __builtin_amdgcn_readfirstlane((unsigned)(uintptr_t)p);
    const unsigned hi = __builtin_amdgcn_readfirstlane((unsigned)((uintptr_t)p >> 32));
    return (const char*)(((uintptr_t)hi << 32) | lo);
}
template <int N> __device__ __forceinline__ void waitvm_c() { asm volatile("s_waitcnt vmcnt(%0)" :: "n"(N) : "memory"); }
template <int PER, int K> __device__ __forceinline__ void waitvm_k(int k) {
    if constexpr (K <= 0) waitvm_c<0>();
    else { if (k >= K) waitvm_c<PER * K>(); else waitvm_k<PER, K - 1>(k); }
}
__device__ __forceinline__ unsigned stage_off(int j, int o) {
    return (unsigned)((j >> 2) * 32768 + (o >> 10) * 4096 + (j & 3) * 1024 + (o & 1023));
}
__device__ __forceinline__ int swzA(int row) { return (-(row >> 2)) & 3; }
__device__ __forceinline__ int swzB(int k) { return ((k >> 1) & 3) | (((k >> 4) & 1) << 2); }

__device__ __forceinline__ bool poll_flags(const char* f, int lane, int n, unsigned call, unsigned* tmo) {
    const unsigned long long t0 = __builtin_amdgcn_s_memrealtime();
    for (unsigned it = 1;; ++it) {
        const unsigned v = ld4_sys_wait(f + lane * 4);
        const bool ok = lane >= n || (int)(v - call) >= 0;
        if (__all(ok)) break;
        if ((it & 15u) == 0u) {
            if (__hip_atomic_load(tmo, __ATOMIC_RELAXED, __HIP_MEMORY_SCOPE_AGENT) == call ||
                __builtin_amdgcn_s_memrealtime() - t0 > DEADLINE) {
                __hip_atomic_store(tmo, call, __ATOMIC_RELAXED, __HIP_MEMORY_SCOPE_AGENT);
                asm volatile("s_waitcnt vmcnt(0)" ::: "memory");
                return false;
            }
        }
        __builtin_amdgcn_s_sleep(1);
    }
    return true;
}

__global__ __launch_bounds__(NTHREADS, 3) void fused_gather_gemm(Params P) {
    __shared__ __attribute__((aligned(1024))) char smem[LDS_BYTES];
    LAS char* L = (LAS char*)smem;
    const int tid = threadIdx.x;
    const int lane = tid & 63;
    const int wid = __builtin_amdgcn_readfirstlane(tid >> 6);
    const int b = blockIdx.x;
    const int rank = P.rank;
    char* wsl = P.ws[rank];
    const unsigned lds0 = (unsigned)(uintptr_t)L;
    unsigned* tmo = (unsigned*)(wsl + WS_TMO);
#define BAR() do { __builtin_amdgcn_sched_barrier(0); __builtin_amdgcn_s_barrier(); __builtin_amdgcn_sched_barrier(0); } while (0)
#define WAITL0() asm volatile("s_waitcnt lgkmcnt(0)" ::: "memory")

    volatile LAS unsigned* mail = (volatile LAS unsigned*)(L + BOFF + NB * BSTG - 16);
    if (wid >= 10) {
        const int lw = wid & 1;
        unsigned voff[4];
#pragma unroll
        for (int i = 0; i < 4; ++i) { const int k = lw * 32 + i * 8 + (lane >> 3); const int ch = (lane & 7) ^ swzB(k);
            voff[i] = (unsigned)(k * NFULL + ch * 16 - i * 1024 + 4096); }
        BAR();
        BAR();
        const int cb = (int)__builtin_amdgcn_readfirstlane(mail[2]) & 15;
        WAITL0();
        BAR();
        const char* gp = uniform_ptr(P.w + (size_t)rank * NSH + cb * 128 - 4096);
        unsigned lp = __builtin_amdgcn_readfirstlane(lds0 + (unsigned)(BOFF + lw * 4096));
        const unsigned lpend = lp + NB * BSTG;
        auto issueB = [&]() {
            unsigned keep;
            asm volatile("s_mov_b32 %0, m0\n\t"
                         "s_mov_b32 m0, %1\n\t"
                         "s_nop 4\n\t"
                         "global_load_lds_dwordx4 %2, %6\n\t"
                         "global_load_lds_dwordx4 %3, %6 offset:1024\n\t"
                         "global_load_lds_dwordx4 %4, %6 offset:2048\n\t"
                         "global_load_lds_dwordx4 %5, %6 offset:3072\n\t"
                         "s_mov_b32 m0, %0"
                         : "=&s"(keep)
                         : "s"(lp), "v"(voff[0]), "v"(voff[1]), "v"(voff[2]), "v"(voff[3]), "s"(gp)
                         : "memory");
            gp += 64u * NFULL;
            lp += BSTG; if (lp == lpend) lp -= NB * BSTG;
        };
#pragma unroll
        for (int t = 0; t < NB; ++t) issueB();
        waitvm_c<4 * (NB - 1)>();
        BAR();
        int issued = NB;
        for (int t = 0; t < NK - 1; ++t) {
            waitvm_k<4, NB - 1>(issued - 2 - t);
            BAR();
            if (t >= SKEW && issued < NK) { issueB(); ++issued; }
        }
        return;
    }
    if (wid >= 8) {
        const int lw = wid & 1;
        const int lag = lw ? SKEW : 0;
        const int npro = DA - lag > 0 ? DA - lag : 0;
        unsigned ticket = 1; int pref = 0;
        GAS unsigned* tick = (GAS unsigned*)(wsl + WS_TICK);
        if (lw == 0) {
            unsigned xcc; asm volatile("s_getreg_b32 %0, hwreg(HW_REG_XCC_ID)" : "=s"(xcc));
            const unsigned cx = 0x6320u, cy = 0x7541u;
            const bool inx = (0x4Du >> rank) & 1u;
            const unsigned own = inx ? cx : cy, oth = inx ? cy : cx;
            pref = (int)((((xcc & 4u) ? oth : own) >> ((xcc & 3u) * 4u)) & 7u);
            if (lane == 0) asm volatile("global_atomic_add %0, %1, %0, off sc0" : "+v"(ticket) : "v"(tick + pref * 32) : "memory");
        }
        BAR();
        const unsigned call = __builtin_amdgcn_readfirstlane(mail[0]);
        WAITL0();
        if (lw == 0) {
            asm volatile("s_waitcnt vmcnt(0)" : "+v"(ticket) :: "memory");
            const unsigned base = (call - 1u) * 32u;
            unsigned t = __builtin_amdgcn_readfirstlane(ticket) - base;
            int s = pref;
            for (int k = 1; t >= 32u && k < NR; ++k) {
                if (lane == 0) __hip_atomic_fetch_sub(tick + s * 32, 1u, __ATOMIC_RELAXED, __HIP_MEMORY_SCOPE_AGENT);
                s = (pref + k) & 7;
                unsigned v = 0;
                if (lane == 0) v = __hip_atomic_fetch_add(tick + s * 32, 1u, __ATOMIC_RELAXED, __HIP_MEMORY_SCOPE_AGENT);
                t = __builtin_amdgcn_readfirstlane(v) - base;
            }
            if (lane == 0) { mail[1] = (unsigned)s; mail[2] = t & 31u; }
            WAITL0();
        }
        BAR();
        const int src = (int)__builtin_amdgcn_readfirstlane(mail[1]) & 7, wi = (int)__builtin_amdgcn_readfirstlane(mail[2]) & 31;
        WAITL0();
        BAR();
        const int rb = wi >> 4;
        const bool local = src == rank;
        bool live = true;
        if (!local && npro > 0) live = poll_flags(wsl + WS_FLAGS + (size_t)src * (NWG * 4) + (rb * 2 + lw) * 256, lane, 8 * ((npro + 7) / 8), call, tmo);
        unsigned voff[8];
        const char* gbase;
        if (local) {
            gbase = P.x + (size_t)(rb * 256 + lw * 128) * KD;
#pragma unroll
            for (int i = 0; i < 8; ++i) { const int row = i * 16 + (lane >> 2); const int ch = (lane & 3) ^ swzA(row);
                voff[i] = (unsigned)(row * KD + ch * 16 - (i & 3) * 1024 + 4096); }
        } else {
            gbase = P.ws[src] + WS_X + (size_t)(call & 1u) * XBYTES + (size_t)(rb * 2 + lw) * XREG;
#pragma unroll
            for (int i = 0; i < 8; ++i) voff[i] = stage_off(0, i * 1024) + (unsigned)(lane * 16 - (i & 3) * 1024 + 4096);
        }
        gbase = uniform_ptr(gbase - 4096);
        int issued = 0;
        unsigned lp = __builtin_amdgcn_readfirstlane(lds0 + (unsigned)(lw * (DA * AUNIT)));
        const unsigned lpend = lp + DA * AUNIT;
        auto issueA = [&]() {
            const char* gp = uniform_ptr(gbase + (local ? (unsigned)(issued * 64) : stage_off(issued, 0)));
            unsigned keep;
            if (live)
            asm volatile("s_mov_b32 %0, m0\n\t"
                         "s_mov_b32 m0, %1\n\t"
                         "s_nop 4\n\t"
                         "global_load_lds_dwordx4 %2, %10\n\t"
                         "global_load_lds_dwordx4 %3, %10 offset:1024\n\t"
                         "global_load_lds_dwordx4 %4, %10 offset:2048\n\t"
                         "global_load_lds_dwordx4 %5, %10 offset:3072\n\t"
                         "s_add_u32 m0, m0, 0x1000\n\t"
                         "s_nop 0\n\t"
                         "global_load_lds_dwordx4 %6, %10\n\t"
                         "global_load_lds_dwordx4 %7, %10 offset:1024\n\t"
                         "global_load_lds_dwordx4 %8, %10 offset:2048\n\t"
                         "global_load_lds_dwordx4 %9, %10 offset:3072\n\t"
                         "s_mov_b32 m0, %0"
                         : "=&s"(keep)
                         : "s"(lp), "v"(voff[0]), "v"(voff[1]), "v"(voff[2]), "v"(voff[3]),
                           "v"(voff[4]), "v"(voff[5]), "v"(voff[6]), "v"(voff[7]), "s"(gp)
                         : "memory", "scc");
            ++issued;
            lp += AUNIT; if (lp == lpend) lp -= DA * AUNIT;
        };
        for (int u = 0; u < npro; ++u) issueA();
        if (lag == 0) waitvm_k<8, DA - 1>(issued - 1);
        BAR();
        for (int t = 0; t < lag + NK - 1; ++t) {
            const int need = t + 1 - lag;
            if (need >= 0) waitvm_k<8, DA - 1>(issued - 1 - need);
            BAR();
            if (issued < NK && issued <= t - lag + DA) issueA();
        }
        return;
    }

    unsigned* ctr = (unsigned*)(wsl + WS_CTR) + b;
    const int xr = (b & 7) * 16 + (tid >> 5), xq = tid & 31, xj = ((b >> 3) & 7) * 8 + (xq >> 2), xc = xq & 3;
    const u32x4 xv = *(const u32x4*)(P.x + (size_t)((b >> 6) * 128 + xr) * KD + xj * 64 + xc * 16);
    const unsigned call = __hip_atomic_load(ctr, __ATOMIC_RELAXED, __HIP_MEMORY_SCOPE_AGENT) + 1u;
    asm volatile("" ::: "memory");
    uintptr_t w0 = (uintptr_t)P.ws[0], w1 = (uintptr_t)P.ws[1], w2 = (uintptr_t)P.ws[2], w3 = (uintptr_t)P.ws[3];
    uintptr_t w4 = (uintptr_t)P.ws[4], w5 = (uintptr_t)P.ws[5], w6 = (uintptr_t)P.ws[6], w7 = (uintptr_t)P.ws[7];
    asm volatile("" : "+s"(w0), "+s"(w1), "+s"(w2), "+s"(w3), "+s"(w4), "+s"(w5), "+s"(w6), "+s"(w7));
    const uintptr_t pw = tid == 0 ? w0 : tid == 1 ? w1 : tid == 2 ? w2 : tid == 3 ? w3 : tid == 4 ? w4 : tid == 5 ? w5 : tid == 6 ? w6 : w7;
    st16_sys(wsl + WS_X + (size_t)(call & 1u) * XBYTES + (size_t)(b >> 6) * XREG + stage_off(xj, xr * 64 + ((xc ^ swzA(xr)) << 4)), xv);
    if (tid == 0) mail[0] = call;
    asm volatile("s_waitcnt vmcnt(0) lgkmcnt(0)" ::: "memory");
    BAR();
    if (tid < NR && tid != rank)
        __hip_atomic_store((GAS unsigned*)((GAS char*)pw + WS_FLAGS) + rank * NWG + b, call, __ATOMIC_RELAXED, __HIP_MEMORY_SCOPE_SYSTEM);
    if (tid == 64) __hip_atomic_store(ctr, call, __ATOMIC_RELAXED, __HIP_MEMORY_SCOPE_AGENT);
    float scale = P.sx[0] * P.sw[0];
    BAR();
    const int src = (int)__builtin_amdgcn_readfirstlane(mail[1]) & 7, wi = (int)__builtin_amdgcn_readfirstlane(mail[2]) & 31;
    WAITL0();
    BAR();
    const int rb = wi >> 4, cb = wi & 15;
    const bool lower = wid >= 4;
    if ((wid & 3) == 1 && src != rank) (void)poll_flags(wsl + WS_FLAGS + (size_t)src * (NWG * 4) + (rb * 2 + (lower ? 1 : 0)) * 256, lane, 64, call, tmo);

    const int wr = wid >> 1, wc = wid & 1, fr = lane & 15, fq = lane >> 4;
    const unsigned aoff = (unsigned)((lower ? DA * AUNIT : 0) + ((wr & 1) * 64 + fr) * 64 + ((fq ^ swzA(fr)) << 4));
    unsigned boff[4];
    {
        const int q = fr >> 1, p = fr & 1, sw = swzB(fq * 16 + q);
#pragma unroll
        for (int n = 0; n < 4; ++n) boff[n] = (unsigned)(BOFF + (fq * 16 + q) * 128 + (((wc * 4 + n) ^ sw) << 4) + p * 8);
    }
    i32x4 acc[4][4];
#pragma unroll
    for (int m = 0; m < 4; ++m)
#pragma unroll
        for (int n = 0; n < 4; ++n) acc[m][n] = (i32x4){0, 0, 0, 0};
    i32x4 af[2][4], bf[2][4];
    auto rd = [&](int h, int sa, int sb) {
        const LAS char* pa = L + sa * AUNIT;
        const LAS char* pb = L + sb * BSTG;
#pragma unroll
        for (int n = 0; n < 4; ++n) {
            const i32x2 lo = __builtin_amdgcn_ds_read_tr8_b64_v2i32((LAS i32x2*)(pb + boff[n]));
            const i32x2 hi = __builtin_amdgcn_ds_read_tr8_b64_v2i32((LAS i32x2*)(pb + boff[n] + 1024));
            bf[h][n] = (i32x4){lo.x, lo.y, hi.x, hi.y};
        }
#pragma unroll
        for (int m = 0; m < 4; ++m) af[h][m] = *(const LAS i32x4*)(pa + aoff + m * 1024);
    };
    auto mm = [&](int h) {
#pragma unroll
        for (int m = 0; m < 4; ++m)
#pragma unroll
            for (int n = 0; n < 4; ++n)
                acc[m][n] = __builtin_amdgcn_mfma_i32_16x16x64_i8(bf[h][n], af[h][m], acc[m][n], 0, 0, 0);
    };
#define INTERLEAVE() do { _Pragma("unroll") for (int _i = 0; _i < 12; ++_i) { \
        __builtin_amdgcn_sched_group_barrier(0x008, 1, 0); __builtin_amdgcn_sched_group_barrier(0x100, 1, 0); } \
        __builtin_amdgcn_sched_group_barrier(0x008, 4, 0); } while (0)
    asm volatile("" : "+v"(scale));
    BAR();
    const unsigned tmo_v = __hip_atomic_load(tmo, __ATOMIC_RELAXED, __HIP_MEMORY_SCOPE_AGENT);
    if (lower) for (int t = 0; t < SKEW; ++t) BAR();
    int sa = 0, sb = 0;
    rd(0, 0, 0);
    for (int u = 0; u < NK; u += 2) {
#pragma unroll
        for (int h = 0; h < 2; ++h) {
            WAITL0();
            BAR();
            sa = (sa + 1 == DA) ? 0 : sa + 1;
            sb = (sb + 1 == NB) ? 0 : sb + 1;
            rd(h ^ 1, sa, sb);
            mm(h);
            INTERLEAVE();
            __builtin_amdgcn_sched_barrier(0);
        }
    }
    {
        int tz = tid; asm volatile("" : "+v"(tz));
        const int ln = tz & 63;
        float* orow = P.out + (size_t)(src * MSH + rb * 256 + wr * 64 + (ln & 15)) * NSH + cb * 128 + wc * 64 + (ln >> 4) * 4;
        unsigned tv = tmo_v; asm volatile("" : "+v"(tv));
        const float sc = (tv == call) ? __builtin_nanf("") : scale;
#pragma unroll
        for (int m = 0; m < 4; ++m)
#pragma unroll
            for (int n = 0; n < 4; ++n) {
                const i32x4 a = acc[m][n];
                const f32x4 v = {(float)a.x * sc, (float)a.y * sc, (float)a.z * sc, (float)a.w * sc};
                st16_out(orow + (size_t)(m * 16) * NSH + n * 16, v);
            }
    }
}
}

extern "C" void kernel_launch(int rank, int nranks, ncclComm_t comm, hipStream_t stream,
                              void* const* d_in, const int* in_sizes, int n_in,
                              void* d_out, int out_size,
                              const int* in_dtypes, int out_dtype,
                              void* const* d_out_all, void* const* d_ws_all, size_t ws_bytes) {
    (void)comm; (void)in_dtypes; (void)out_dtype; (void)d_out_all;
    if (nranks != ag::NR || n_in != 4 || (unsigned)rank >= (unsigned)nranks || ws_bytes < ag::WS_NEED ||
        in_sizes[0] != ag::MSH * ag::KD || in_sizes[1] != ag::KD * ag::NFULL || out_size != ag::NR * ag::MSH * ag::NSH) {
        fprintf(stderr, "kernel_launch: unexpected problem shape\n");
        return;
    }
    ag::Params p;
    p.x = (const char*)d_in[0]; p.w = (const char*)d_in[1];
    p.sx = (const float*)d_in[2]; p.sw = (const float*)d_in[3];
    p.out = (float*)d_out;
    for (int r = 0; r < ag::NR; ++r) p.ws[r] = (char*)d_ws_all[r];
    p.rank = rank;
    hipLaunchKernelGGL(ag::fused_gather_gemm, dim3(ag::NWG), dim3(ag::NTHREADS), 0, stream, p);
    HIP_CHECK(hipGetLastError());
}
```

```cpp
#include <hip/hip_runtime.h>
#include <rccl/rccl.h>
#include <cstdio>
#include <cstdint>

#define HIP_CHECK(expr) do { hipError_t _e = (expr); if (_e != hipSuccess) { \
    fprintf(stderr, "HIP error %s at %s:%d\n", hipGetErrorString(_e), __FILE__, __LINE__); \
    return; } } while (0)

#define LAS __attribute__((address_space(3)))
#define GAS __attribute__((address_space(1)))
typedef int      i32x4 __attribute__((ext_vector_type(4)));
typedef int      i32x2 __attribute__((ext_vector_type(2)));
typedef unsigned u32x4 __attribute__((ext_vector_type(4)));
typedef float    f32x4 __attribute__((ext_vector_type(4)));

namespace ag {
constexpr int NR = 8;
constexpr int MSH = 512;
constexpr int KD = 4096;
constexpr int NFULL = 16384;
constexpr int NSH = 2048;
constexpr int NWG = 256;
constexpr int AUNIT = 8192;
constexpr int BSTG = 8192;
#define AG_DAU 5
#define AG_DAL 5
#define AG_NB 10
#define AG_S 5
constexpr int DAU = AG_DAU, DAL = AG_DAL;
constexpr int DAM = DAU > DAL ? DAU : DAL;
constexpr int NB = AG_NB;
constexpr int SKEW = AG_S;
constexpr int NK = 64;
constexpr int BOFF = (DAU + DAL) * AUNIT;
constexpr int LDS_BYTES = BOFF + NB * BSTG;
constexpr int NTHREADS = 768;
constexpr size_t XREG = (size_t)NK * AUNIT;
constexpr size_t XBYTES = 4 * XREG;
constexpr size_t WS_CTR = 0;
constexpr size_t WS_TMO = 4096;
constexpr size_t WS_TICK = 4096 + 1024;
constexpr size_t WS_FLAGS = 8192;
constexpr size_t WS_X = 1u << 20;
constexpr size_t WS_NEED = WS_X + 2 * XBYTES;
constexpr unsigned long long DEADLINE = 50000000ull;
static_assert(DAU >= 3 && DAL >= 3 && DAM <= 8 && SKEW >= 0 && NB - SKEW >= 3 && NB - 1 <= 15 && LDS_BYTES <= 163840, "layout");

struct Params {
    const char* x; const char* w; const float* sx; const float* sw; float* out;
    char* ws[NR];
    int rank;
};

__device__ __forceinline__ void st16_sys(void* p, u32x4 v) {
    asm volatile("global_store_dwordx4 %0, %1, off sc0 sc1\n\ts_nop 1" :: "v"(p), "v"(v) : "memory");
}
__device__ __forceinline__ void st16_out(float* p, f32x4 v) {
    asm volatile("global_store_dwordx4 %0, %1, off sc1\n\ts_nop 1" :: "v"(p), "v"(v) : "memory");
}
__device__ __forceinline__ unsigned ld4_sys_wait(const void* p) {
    unsigned v;
    asm volatile("global_load_dword %0, %1, off sc0 sc1\n\ts_waitcnt vmcnt(0)" : "=&v"(v) : "v"(p) : "memory");
    return v;
}
__device__ __forceinline__ const char* uniform_ptr(const char* p) {
    const unsigned lo = __builtin_amdgcn_readfirstlane((unsigned)(uintptr_t)p);
    const unsigned hi = __builtin_amdgcn_readfirstlane((unsigned)((uintptr_t)p >> 32));
    return (const char*)(((uintptr_t)hi << 32) | lo);
}
template <int N> __device__ __forceinline__ void waitvm_c() { asm volatile("s_waitcnt vmcnt(%0)" :: "n"(N) : "memory"); }
template <int PER, int K> __device__ __forceinline__ void waitvm_k(int k) {
    if constexpr (K <= 0) waitvm_c<0>();
    else { if (k >= K) waitvm_c<PER * K>(); else waitvm_k<PER, K - 1>(k); }
}
__device__ __forceinline__ unsigned stage_off(int j, int o) {
    return (unsigned)((j >> 2) * 32768 + (o >> 10) * 4096 + (j & 3) * 1024 + (o & 1023));
}
__device__ __forceinline__ int swzA(int row) { return (-(row >> 2)) & 3; }
__device__ __forceinline__ int swzB(int k) { return ((k >> 1) & 3) | (((k >> 4) & 1) << 2); }

__device__ __forceinline__ bool poll_flags(const char* f, int lane, int n, unsigned call, unsigned* tmo) {
    const unsigned long long t0 = __builtin_amdgcn_s_memrealtime();
    for (unsigned it = 1;; ++it) {
        const unsigned v = ld4_sys_wait(f + lane * 4);
        const bool ok = lane >= n || (int)(v - call) >= 0;
        if (__all(ok)) break;
        if ((it & 15u) == 0u) {
            if (__hip_atomic_load(tmo, __ATOMIC_RELAXED, __HIP_MEMORY_SCOPE_AGENT) == call ||
                __builtin_amdgcn_s_memrealtime() - t0 > DEADLINE) {
                __hip_atomic_store(tmo, call, __ATOMIC_RELAXED, __HIP_MEMORY_SCOPE_AGENT);
                asm volatile("s_waitcnt vmcnt(0)" ::: "memory");
                return false;
            }
        }
        __builtin_amdgcn_s_sleep(1);
    }
    return true;
}

__global__ __launch_bounds__(NTHREADS, 3) void fused_gather_gemm(Params P) {
    __shared__ __attribute__((aligned(1024))) char smem[LDS_BYTES];
    LAS char* L = (LAS char*)smem;
    const int tid = threadIdx.x;
    const int lane = tid & 63;
    const int wid = __builtin_amdgcn_readfirstlane(tid >> 6);
    const int b = blockIdx.x;
    const int rank = P.rank;
    char* wsl = P.ws[rank];
    const unsigned lds0 = (unsigned)(uintptr_t)L;
    unsigned* tmo = (unsigned*)(wsl + WS_TMO);
#define BAR() do { __builtin_amdgcn_sched_barrier(0); __builtin_amdgcn_s_barrier(); __builtin_amdgcn_sched_barrier(0); } while (0)
#define WAITL0() asm volatile("s_waitcnt lgkmcnt(0)" ::: "memory")

    volatile LAS unsigned* mail = (volatile LAS unsigned*)(L + BOFF + NB * BSTG - 16);
    if (wid >= 10) {
        const int lw = wid & 1;
        unsigned voff[4];
#pragma unroll
        for (int i = 0; i < 4; ++i) { const int k = lw * 32 + i * 8 + (lane >> 3); const int ch = (lane & 7) ^ swzB(k);
            voff[i] = (unsigned)(k * NFULL + ch * 16 - i * 1024 + 4096); }
        BAR();
        BAR();
        const int cb = (int)__builtin_amdgcn_readfirstlane(mail[2]) & 15;
        WAITL0();
        BAR();
        const char* gp = uniform_ptr(P.w + (size_t)rank * NSH + cb * 128 - 4096);
        unsigned lp = __builtin_amdgcn_readfirstlane(lds0 + (unsigned)(BOFF + lw * 4096));
        const unsigned lpend = lp + NB * BSTG;
        auto issueB = [&]() {
            unsigned keep;
            asm volatile("s_mov_b32 %0, m0\n\t"
                         "s_mov_b32 m0, %1\n\t"
                         "s_nop 4\n\t"
                         "global_load_lds_dwordx4 %2, %6\n\t"
                         "global_load_lds_dwordx4 %3, %6 offset:1024\n\t"
                         "global_load_lds_dwordx4 %4, %6 offset:2048\n\t"
                         "global_load_lds_dwordx4 %5, %6 offset:3072\n\t"
                         "s_mov_b32 m0, %0"
                         : "=&s"(keep)
                         : "s"(lp), "v"(voff[0]), "v"(voff[1]), "v"(voff[2]), "v"(voff[3]), "s"(gp)
                         : "memory");
            gp += 64u * NFULL;
            lp += BSTG; if (lp == lpend) lp -= NB * BSTG;
        };
#pragma unroll
        for (int t = 0; t < NB; ++t) issueB();
        waitvm_c<4 * (NB - 1)>();
        BAR();
        int issued = NB;
        for (int t = 0; t < NK - 1; ++t) {
            waitvm_k<4, NB - 1>(issued - 2 - t);
            BAR();
            if (t >= SKEW && issued < NK) { issueB(); ++issued; }
        }
        return;
    }
    if (wid >= 8) {
        const int lw = wid & 1;
        const int lag = lw ? SKEW : 0;
        const int da = lw ? DAL : DAU;
        const int npro = da - lag > 0 ? da - lag : 0;
        unsigned ticket = 1; int pref = 0;
        GAS unsigned* tick = (GAS unsigned*)(wsl + WS_TICK);
        if (lw == 0) {
            unsigned xcc; asm volatile("s_getreg_b32 %0, hwreg(HW_REG_XCC_ID)" : "=s"(xcc));
            const unsigned cx = 0x6320u, cy = 0x7541u;
            const bool inx = (0x4Du >> rank) & 1u;
            const unsigned own = inx ? cx : cy, oth = inx ? cy : cx;
            pref = (int)((((xcc & 4u) ? oth : own) >> ((xcc & 3u) * 4u)) & 7u);
            if (lane == 0) asm volatile("global_atomic_add %0, %1, %0, off sc0" : "+v"(ticket) : "v"(tick + pref * 32) : "memory");
        }
        BAR();
        const unsigned call = __builtin_amdgcn_readfirstlane(mail[0]);
        WAITL0();
        if (lw == 0) {
            asm volatile("s_waitcnt vmcnt(0)" : "+v"(ticket) :: "memory");
            const unsigned base = (call - 1u) * 32u;
            unsigned t = __builtin_amdgcn_readfirstlane(ticket) - base;
            int s = pref;
            for (int k = 1; t >= 32u && k < NR; ++k) {
                if (lane == 0) __hip_atomic_fetch_sub(tick + s * 32, 1u, __ATOMIC_RELAXED, __HIP_MEMORY_SCOPE_AGENT);
                s = (pref + k) & 7;
                unsigned v = 0;
                if (lane == 0) v = __hip_atomic_fetch_add(tick + s * 32, 1u, __ATOMIC_RELAXED, __HIP_MEMORY_SCOPE_AGENT);
                t = __builtin_amdgcn_readfirstlane(v) - base;
            }
            if (lane == 0) { mail[1] = (unsigned)s; mail[2] = t & 31u; }
            WAITL0();
        }
        BAR();
        const int src = (int)__builtin_amdgcn_readfirstlane(mail[1]) & 7, wi = (int)__builtin_amdgcn_readfirstlane(mail[2]) & 31;
        WAITL0();
        BAR();
        const int rb = wi >> 4;
        const bool local = src == rank;
        bool live = true;
        unsigned voff[8];
        const char* gbase;
        if (local) {
            gbase = P.x + (size_t)(rb * 256 + lw * 128) * KD;
#pragma unroll
            for (int i = 0; i < 8; ++i) { const int row = i * 16 + (lane >> 2); const int ch = (lane & 3) ^ swzA(row);
                voff[i] = (unsigned)(row * KD + ch * 16 - (i & 3) * 1024 + 4096); }
        } else {
            gbase = P.ws[src] + WS_X + (size_t)(call & 1u) * XBYTES + (size_t)(rb * 2 + lw) * XREG;
#pragma unroll
            for (int i = 0; i < 8; ++i) voff[i] = stage_off(0, i * 1024) + (unsigned)(lane * 16 - (i & 3) * 1024 + 4096);
        }
        gbase = uniform_ptr(gbase - 4096);
        int issued = 0;
        unsigned lp = __builtin_amdgcn_readfirstlane(lds0 + (unsigned)(lw * (DAU * AUNIT)));
        const unsigned lpend = lp + da * AUNIT, lpbeg = lp;
        auto issueA = [&]() {
            const char* gp = uniform_ptr(gbase + (local ? (unsigned)(issued * 64) : stage_off(issued, 0)));
            unsigned keep;
            if (live)
            asm volatile("s_mov_b32 %0, m0\n\t"
                         "s_mov_b32 m0, %1\n\t"
                         "s_nop 4\n\t"
                         "global_load_lds_dwordx4 %2, %10\n\t"
                         "global_load_lds_dwordx4 %3, %10 offset:1024\n\t"
                         "global_load_lds_dwordx4 %4, %10 offset:2048\n\t"
                         "global_load_lds_dwordx4 %5, %10 offset:3072\n\t"
                         "s_add_u32 m0, m0, 0x1000\n\t"
                         "s_nop 0\n\t"
                         "global_load_lds_dwordx4 %6, %10\n\t"
                         "global_load_lds_dwordx4 %7, %10 offset:1024\n\t"
                         "global_load_lds_dwordx4 %8, %10 offset:2048\n\t"
                         "global_load_lds_dwordx4 %9, %10 offset:3072\n\t"
                         "s_mov_b32 m0, %0"
                         : "=&s"(keep)
                         : "s"(lp), "v"(voff[0]), "v"(voff[1]), "v"(voff[2]), "v"(voff[3]),
                           "v"(voff[4]), "v"(voff[5]), "v"(voff[6]), "v"(voff[7]), "s"(gp)
                         : "memory", "scc");
            ++issued;
            lp += AUNIT; if (lp == lpend) lp = lpbeg;
        };
        asm volatile("" : "+v"(voff[0]), "+v"(voff[1]), "+v"(voff[2]), "+v"(voff[3]), "+v"(voff[4]), "+v"(voff[5]), "+v"(voff[6]), "+v"(voff[7]));
        if (!local && npro > 0) live = poll_flags(wsl + WS_FLAGS + (size_t)src * (NWG * 4) + (rb * 2 + lw) * 256, lane, 8 * ((npro + 7) / 8), call, tmo);
        for (int u = 0; u < npro; ++u) issueA();
        if (lag == 0) waitvm_k<8, DAM - 1>(issued - 1);
        BAR();
        for (int t = 0; t < lag + NK - 1; ++t) {
            const int need = t + 1 - lag;
            if (need >= 0) waitvm_k<8, DAM - 1>(issued - 1 - need);
            BAR();
            if (issued < NK && issued <= t - lag + da) issueA();
        }
        return;
    }

    unsigned* ctr = (unsigned*)(wsl + WS_CTR) + b;
    const int xr = (b & 7) * 16 + (tid >> 5), xq = tid & 31, xj = ((b >> 3) & 7) * 8 + (xq >> 2), xc = xq & 3;
    const u32x4 xv = *(const u32x4*)(P.x + (size_t)((b >> 6) * 128 + xr) * KD + xj * 64 + xc * 16);
    const unsigned call = __hip_atomic_load(ctr, __ATOMIC_RELAXED, __HIP_MEMORY_SCOPE_AGENT) + 1u;
    asm volatile("" ::: "memory");
    uintptr_t w0 = (uintptr_t)P.ws[0], w1 = (uintptr_t)P.ws[1], w2 = (uintptr_t)P.ws[2], w3 = (uintptr_t)P.ws[3];
    uintptr_t w4 = (uintptr_t)P.ws[4], w5 = (uintptr_t)P.ws[5], w6 = (uintptr_t)P.ws[6], w7 = (uintptr_t)P.ws[7];
    asm volatile("" : "+s"(w0), "+s"(w1), "+s"(w2), "+s"(w3), "+s"(w4), "+s"(w5), "+s"(w6), "+s"(w7));
    const uintptr_t pw = tid == 0 ? w0 : tid == 1 ? w1 : tid == 2 ? w2 : tid == 3 ? w3 : tid == 4 ? w4 : tid == 5 ? w5 : tid == 6 ? w6 : w7;
    st16_sys(wsl + WS_X + (size_t)(call & 1u) * XBYTES + (size_t)(b >> 6) * XREG + stage_off(xj, xr * 64 + ((xc ^ swzA(xr)) << 4)), xv);
    if (tid == 0) mail[0] = call;
    asm volatile("s_waitcnt vmcnt(0) lgkmcnt(0)" ::: "memory");
    BAR();
    if (tid < NR && tid != rank)
        __hip_atomic_store((GAS unsigned*)((GAS char*)pw + WS_FLAGS) + rank * NWG + b, call, __ATOMIC_RELAXED, __HIP_MEMORY_SCOPE_SYSTEM);
    if (tid == 64) __hip_atomic_store(ctr, call, __ATOMIC_RELAXED, __HIP_MEMORY_SCOPE_AGENT);
    float scale = P.sx[0] * P.sw[0];
    BAR();
    const int src = (int)__builtin_amdgcn_readfirstlane(mail[1]) & 7, wi = (int)__builtin_amdgcn_readfirstlane(mail[2]) & 31;
    WAITL0();
    BAR();
    const int rb = wi >> 4, cb = wi & 15;
    const bool lower = wid >= 4;
    if ((wid & 3) == 1 && src != rank) (void)poll_flags(wsl + WS_FLAGS + (size_t)src * (NWG * 4) + (rb * 2 + (lower ? 1 : 0)) * 256, lane, 64, call, tmo);

    const int wr = wid >> 1, wc = wid & 1, fr = lane & 15, fq = lane >> 4;
    const unsigned aoff = (unsigned)((lower ? DAU * AUNIT : 0) + ((wr & 1) * 64 + fr) * 64 + ((fq ^ swzA(fr)) << 4));
    unsigned boff[4];
    {
        const int q = fr >> 1, p = fr & 1, sw = swzB(fq * 16 + q);
#pragma unroll
        for (int n = 0; n < 4; ++n) boff[n] = (unsigned)(BOFF + (fq * 16 + q) * 128 + (((wc * 4 + n) ^ sw) << 4) + p * 8);
    }
    i32x4 acc[4][4];
#pragma unroll
    for (int m = 0; m < 4; ++m)
#pragma unroll
        for (int n = 0; n < 4; ++n) acc[m][n] = (i32x4){0, 0, 0, 0};
    i32x4 af[2][4], bf[2][4];
    auto rd = [&](int h, int sa, int sb) {
        const LAS char* pa = L + sa * AUNIT;
        const LAS char* pb = L + sb * BSTG;
#pragma unroll
        for (int n = 0; n < 4; ++n) {
            const i32x2 lo = __builtin_amdgcn_ds_read_tr8_b64_v2i32((LAS i32x2*)(pb + boff[n]));
            const i32x2 hi = __builtin_amdgcn_ds_read_tr8_b64_v2i32((LAS i32x2*)(pb + boff[n] + 1024));
            bf[h][n] = (i32x4){lo.x, lo.y, hi.x, hi.y};
        }
#pragma unroll
        for (int m = 0; m < 4; ++m) af[h][m] = *(const LAS i32x4*)(pa + aoff + m * 1024);
    };
    auto mm = [&](int h) {
#pragma unroll
        for (int m = 0; m < 4; ++m)
#pragma unroll
            for (int n = 0; n < 4; ++n)
                acc[m][n] = __builtin_amdgcn_mfma_i32_16x16x64_i8(bf[h][n], af[h][m], acc[m][n], 0, 0, 0);
    };
#define INTERLEAVE() do { _Pragma("unroll") for (int _i = 0; _i < 12; ++_i) { \
        __builtin_amdgcn_sched_group_barrier(0x008, 1, 0); __builtin_amdgcn_sched_group_barrier(0x100, 1, 0); } \
        __builtin_amdgcn_sched_group_barrier(0x008, 4, 0); } while (0)
    asm volatile("" : "+v"(scale));
    BAR();
    const unsigned tmo_v = __hip_atomic_load(tmo, __ATOMIC_RELAXED, __HIP_MEMORY_SCOPE_AGENT);
    if (lower) for (int t = 0; t < SKEW; ++t) BAR();
    const int da = lower ? DAL : DAU;
    int sa = 0, sb = 0;
    rd(0, 0, 0);
    for (int u = 0; u < NK; u += 2) {
#pragma unroll
        for (int h = 0; h < 2; ++h) {
            WAITL0();
            BAR();
            sa = (sa + 1 == da) ? 0 : sa + 1;
            sb = (sb + 1 == NB) ? 0 : sb + 1;
            rd(h ^ 1, sa, sb);
            mm(h);
            INTERLEAVE();
            __builtin_amdgcn_sched_barrier(0);
        }
    }
    {
        int tz = tid; asm volatile("" : "+v"(tz));
        const int ln = tz & 63;
        float* orow = P.out + (size_t)(src * MSH + rb * 256 + wr * 64 + (ln & 15)) * NSH + cb * 128 + wc * 64 + (ln >> 4) * 4;
        unsigned tv = tmo_v; asm volatile("" : "+v"(tv));
        const float sc = (tv == call) ? __builtin_nanf("") : scale;
#pragma unroll
        for (int m = 0; m < 4; ++m)
#pragma unroll
            for (int n = 0; n < 4; ++n) {
                const i32x4 a = acc[m][n];
                const f32x4 v = {(float)a.x * sc, (float)a.y * sc, (float)a.z * sc, (float)a.w * sc};
                st16_out(orow + (size_t)(m * 16) * NSH + n * 16, v);
            }
    }
}
}

extern "C" void kernel_launch(int rank, int nranks, ncclComm_t comm, hipStream_t stream,
                              void* const* d_in, const int* in_sizes, int n_in,
                              void* d_out, int out_size,
                              const int* in_dtypes, int out_dtype,
                              void* const* d_out_all, void* const* d_ws_all, size_t ws_bytes) {
    (void)comm; (void)in_dtypes; (void)out_dtype; (void)d_out_all;
    if (nranks != ag::NR || n_in != 4 || (unsigned)rank >= (unsigned)nranks || ws_bytes < ag::WS_NEED ||
        in_sizes[0] != ag::MSH * ag::KD || in_sizes[1] != ag::KD * ag::NFULL || out_size != ag::NR * ag::MSH * ag::NSH) {
        fprintf(stderr, "kernel_launch: unexpected problem shape\n");
        return;
    }
    ag::Params p;
    p.x = (const char*)d_in[0]; p.w = (const char*)d_in[1];
    p.sx = (const float*)d_in[2]; p.sw = (const float*)d_in[3];
    p.out = (float*)d_out;
    for (int r = 0; r < ag::NR; ++r) p.ws[r] = (char*)d_ws_all[r];
    p.rank = rank;
    hipLaunchKernelGGL(ag::fused_gather_gemm, dim3(ag::NWG), dim3(ag::NTHREADS), 0, stream, p);
    HIP_CHECK(hipGetLastError());
}
```
